```python
import numpy as np

kernel_name = "gpu_gemm_rs_m2048_k2048_n2048_f32_none"


ABSMAX_RTOL = 0.001

COMPUTE_DTYPE = "float32"
EPILOGUE = "none"
IN_QUANT = None
OUT_QUANT = None
IN_DTYPES = {"x": "float32", "w_mat": "float32"}
WORLD_SIZE = 8

def setup_inputs():

    x = np.random.randn(2048, 2048).astype(np.float32)
    w_mat = (np.random.randn(2048, 2048) / np.sqrt(2048)).astype(np.float32)
    return {"x": x, "w_mat": w_mat}

def _epilogue(y):
    return y

def reference(x, w_mat):
    y = (x @ w_mat).astype(np.float32)
    return _epilogue(y)
```

```cpp
#include <hip/hip_runtime.h>
#include <rccl/rccl.h>
#include <cstdio>
#include <cstdint>

#define HIP_CHECK(expr) do { hipError_t _e = (expr); if (_e != hipSuccess) { \
    fprintf(stderr, "HIP error %s at %s:%d\n", hipGetErrorString(_e), __FILE__, __LINE__); \
    return; } } while (0)

#define ST_AUX 17

namespace {

typedef _Float16 half8 __attribute__((ext_vector_type(8)));
typedef float f32x4 __attribute__((ext_vector_type(4)));
typedef unsigned u32x4 __attribute__((ext_vector_type(4)));
typedef unsigned long long u64;

constexpr int NR = 8;
constexpr int ND = 2048;
constexpr int MS = 256;
constexpr int KS = 256;
constexpr int NWG = 256;
constexpr int NTHREADS = 576;

constexpr unsigned OFF_CTRP = 0;
constexpr unsigned OFF_CTRC = 32768;
constexpr unsigned OFF_TMO = 65536;
constexpr unsigned OFF_FLAGS = 131072;
constexpr unsigned FW_BYTES = 8 * 4 * 32 * 128;
constexpr unsigned FX_BYTES = 8 * 8 * 4 * 128;
constexpr unsigned FL_BYTES = FW_BYTES + FX_BYTES;
constexpr unsigned OFF_DATA = 1u << 20;
constexpr unsigned WF_BYTES = 8u << 20;
constexpr unsigned XF_BYTES = 1u << 20;
constexpr unsigned DATA_BYTES = WF_BYTES + XF_BYTES;
constexpr unsigned WS_NEED = OFF_DATA + 2 * DATA_BYTES;
constexpr u64 DEADLINE_TICKS = 50000000ull;

struct Params {
    const float* x;
    const float* w;
    float* out;
    char* ws[NR];
    int rank;
};

__device__ __forceinline__ unsigned wave_umax(unsigned v) {
#pragma unroll
    for (int o = 32; o; o >>= 1) {
        unsigned t = (unsigned)__shfl_xor((int)v, o);
        v = v > t ? v : t;
    }
    return v;
}

__device__ __forceinline__ unsigned absbits(float v) { return __builtin_bit_cast(unsigned, v) & 0x7fffffffu; }
__device__ __forceinline__ unsigned umax4(unsigned m, f32x4 v) {
    unsigned a = absbits(v[0]), b = absbits(v[1]), c = absbits(v[2]), d = absbits(v[3]);
    a = a > b ? a : b; c = c > d ? c : d; a = a > c ? a : c;
    return m > a ? m : a;
}
__device__ __forceinline__ unsigned block_exp(unsigned lane_max) {
    unsigned e = __builtin_amdgcn_readfirstlane(wave_umax(lane_max)) >> 23;
    e = e < 1u ? 1u : e;
    e = e > 253u ? 253u : e;
    return e;
}

__device__ __forceinline__ void flag_store(char* base, unsigned off, unsigned call, unsigned e) {
    __hip_atomic_store((u64*)(base + off), ((u64)call << 32) | (u64)e, __ATOMIC_RELAXED, __HIP_MEMORY_SCOPE_SYSTEM);
}

__device__ __forceinline__ unsigned wait_flag(const char* base, unsigned off, unsigned call, unsigned* tmo, bool& ok) {
    const u64* f = (const u64*)(base + off);
    u64 v = __hip_atomic_load(f, __ATOMIC_RELAXED, __HIP_MEMORY_SCOPE_SYSTEM);
    unsigned hi = __builtin_amdgcn_readfirstlane((unsigned)(v >> 32));
    if (hi < call) {
        const u64 t0 = __builtin_amdgcn_s_memrealtime();
        for (;;) {
            __builtin_amdgcn_s_sleep(2);
            v = __hip_atomic_load(f, __ATOMIC_RELAXED, __HIP_MEMORY_SCOPE_SYSTEM);
            hi = __builtin_amdgcn_readfirstlane((unsigned)(v >> 32));
            if (hi >= call) break;
            unsigned t = __builtin_amdgcn_readfirstlane(__hip_atomic_load(tmo, __ATOMIC_RELAXED, __HIP_MEMORY_SCOPE_AGENT));
            if (t == call || __builtin_amdgcn_s_memrealtime() - t0 > DEADLINE_TICKS) {
                __hip_atomic_store(tmo, call, __ATOMIC_RELAXED, __HIP_MEMORY_SCOPE_AGENT);
                ok = false;
                break;
            }
        }
    }
    asm volatile("" ::: "memory");
    return __builtin_amdgcn_readfirstlane((unsigned)v);
}

__device__ __forceinline__ u32x4 pack8(float a0, float a1, float a2, float a3, float a4, float a5, float a6, float a7, float m) {
    half8 h;
    h[0] = (_Float16)(a0 * m); h[1] = (_Float16)(a1 * m); h[2] = (_Float16)(a2 * m); h[3] = (_Float16)(a3 * m);
    h[4] = (_Float16)(a4 * m); h[5] = (_Float16)(a5 * m); h[6] = (_Float16)(a6 * m); h[7] = (_Float16)(a7 * m);
    return __builtin_bit_cast(u32x4, h);
}

__global__ void __launch_bounds__(NTHREADS) gemm_ag_kernel(Params P) {
    __shared__ __attribute__((aligned(16))) float red[8][32][64];
    __shared__ int s_bad;

    const int g = blockIdx.x;
    const int wave = __builtin_amdgcn_readfirstlane(threadIdx.x >> 6);
    const int lane = threadIdx.x & 63;
    const int lg = lane >> 4, lc = lane & 15;
    const int rank = P.rank;
    char* const myws = P.ws[rank];

    if (wave == 8) {
        const int p = (g >> 3) & 7;
        const int b = (g & 7) * 4 + (g >> 6);
        unsigned* ctr = (unsigned*)(myws + OFF_CTRP + g * 128);
        const unsigned call = __builtin_amdgcn_readfirstlane(__hip_atomic_load(ctr, __ATOMIC_RELAXED, __HIP_MEMORY_SCOPE_AGENT)) + 1u;

        const int mblk = b & 7, cx = b >> 3;
        f32x4 xv[2][2][2];
#pragma unroll
        for (int ks = 0; ks < 2; ++ks)
#pragma unroll
            for (int rb = 0; rb < 2; ++rb) {
                const float* src = P.x + (size_t)(p * MS + mblk * 32 + rb * 16 + lc) * KS + cx * 64 + ks * 32 + lg * 8;
                xv[ks][rb][0] = *(const f32x4*)src;
                xv[ks][rb][1] = *(const f32x4*)(src + 4);
            }
        const float* wsrc = P.w + (size_t)(lg * 8) * ND + b * 64 + lc * 4;
        f32x4 wa[2][8], wb[2][8];
#define LOAD_W(dst, c) _Pragma("unroll") for (int ks = 0; ks < 2; ++ks) _Pragma("unroll") for (int j = 0; j < 8; ++j) \
            dst[ks][j] = *(const f32x4*)(wsrc + (size_t)((c) * 64 + ks * 32 + j) * ND);
        LOAD_W(wa, 0)

        char* const dws = P.ws[p];
        const unsigned par = call & 1u;
        const __amdgpu_buffer_rsrc_t rs = __builtin_amdgcn_make_buffer_rsrc(dws + OFF_DATA + par * DATA_BYTES, 0, (int)DATA_BYTES, 0x00020000);
        const unsigned fl = OFF_FLAGS + par * FL_BYTES;

        unsigned ex;
        {
            unsigned m = 0;
#pragma unroll
            for (int ks = 0; ks < 2; ++ks)
#pragma unroll
                for (int rb = 0; rb < 2; ++rb) { m = umax4(m, xv[ks][rb][0]); m = umax4(m, xv[ks][rb][1]); }
            ex = block_exp(m);
            const float mul = __builtin_bit_cast(float, (254u - ex) << 23);
            const unsigned off = WF_BYTES + (unsigned)(((rank * 8 + mblk) * 4 + cx) * 4) * 1024u + lane * 16u;
#pragma unroll
            for (int ks = 0; ks < 2; ++ks)
#pragma unroll
                for (int rb = 0; rb < 2; ++rb) {
                    const f32x4 a = xv[ks][rb][0], c = xv[ks][rb][1];
                    __builtin_amdgcn_raw_buffer_store_b128(pack8(a[0], a[1], a[2], a[3], c[0], c[1], c[2], c[3], mul), rs,
                                                           off + (ks * 2 + rb) * 1024u, 0, ST_AUX);
                }
        }
        unsigned ew[4];
#define PUSH_W(src, c) { \
            unsigned m = 0; \
            _Pragma("unroll") for (int ks = 0; ks < 2; ++ks) _Pragma("unroll") for (int j = 0; j < 8; ++j) m = umax4(m, src[ks][j]); \
            ew[c] = block_exp(m); \
            const float mul = __builtin_bit_cast(float, (254u - ew[c]) << 23); \
            const unsigned off = (unsigned)((rank * 4 + (c)) * 32 + b) * 8192u + lane * 16u; \
            _Pragma("unroll") for (int ks = 0; ks < 2; ++ks) _Pragma("unroll") for (int i = 0; i < 4; ++i) \
                __builtin_amdgcn_raw_buffer_store_b128(pack8(src[ks][0][i], src[ks][1][i], src[ks][2][i], src[ks][3][i], \
                                                             src[ks][4][i], src[ks][5][i], src[ks][6][i], src[ks][7][i], mul), \
                                                       rs, off + (ks * 4 + i) * 1024u, 0, ST_AUX); \
        }
#define FW_OFF(c) (fl + (unsigned)((rank * 4 + (c)) * 32 + b) * 128u)
        asm volatile("" ::: "memory");
        LOAD_W(wb, 1)
        PUSH_W(wa, 0)
        asm volatile("" ::: "memory");
        LOAD_W(wa, 2)
        asm volatile("s_waitcnt vmcnt(24)" ::: "memory");
        if (lane == 0) flag_store(dws, fl + FW_BYTES + (unsigned)((rank * 8 + mblk) * 4 + cx) * 128u, call, ex);
        PUSH_W(wb, 1)
        asm volatile("" ::: "memory");
        LOAD_W(wb, 3)
        asm volatile("s_waitcnt vmcnt(25)" ::: "memory");
        if (lane == 0) flag_store(dws, FW_OFF(0), call, ew[0]);
        PUSH_W(wa, 2)
        asm volatile("s_waitcnt vmcnt(9)" ::: "memory");
        if (lane == 0) flag_store(dws, FW_OFF(1), call, ew[1]);
        PUSH_W(wb, 3)
        asm volatile("s_waitcnt vmcnt(9)" ::: "memory");
        if (lane == 0) flag_store(dws, FW_OFF(2), call, ew[2]);
        asm volatile("s_waitcnt vmcnt(1)" ::: "memory");
        if (lane == 0) {
            flag_store(dws, FW_OFF(3), call, ew[3]);
            __hip_atomic_store(ctr, call, __ATOMIC_RELAXED, __HIP_MEMORY_SCOPE_AGENT);
        }
        return;
    }

    const int j = wave;
    const int t = g >> 3;
    const int nblk = (g & 7) * 4 + (t & 3);
    const int mblk = t >> 2;
    unsigned* ctr = (unsigned*)(myws + OFF_CTRC + g * 128);
    unsigned* tmo = (unsigned*)(myws + OFF_TMO);
    const unsigned call = __builtin_amdgcn_readfirstlane(__hip_atomic_load(ctr, __ATOMIC_RELAXED, __HIP_MEMORY_SCOPE_AGENT)) + 1u;
    const unsigned par = call & 1u;
    const unsigned fl = OFF_FLAGS + par * FL_BYTES;
    const char* data = myws + OFF_DATA + par * DATA_BYTES;
    if (threadIdx.x == 0) s_bad = 0;

    bool ok = true;
    unsigned ex[4];
#pragma unroll
    for (int c = 0; c < 4; ++c)
        ex[c] = wait_flag(myws, fl + FW_BYTES + (unsigned)((j * 8 + mblk) * 4 + c) * 128u, call, tmo, ok);

    f32x4 acc[2][4];
#pragma unroll
    for (int rb = 0; rb < 2; ++rb)
#pragma unroll
        for (int i = 0; i < 4; ++i) acc[rb][i] = (f32x4){0.f, 0.f, 0.f, 0.f};

    const char* xa = data + WF_BYTES + (size_t)((j * 8 + mblk) * 16) * 1024 + lane * 16;
#pragma unroll
    for (int c = 0; c < 4; ++c) {
        half8 a[2][2], bf[2][4];
#pragma unroll
        for (int ks = 0; ks < 2; ++ks)
#pragma unroll
            for (int rb = 0; rb < 2; ++rb) a[ks][rb] = *(const half8*)(xa + (c * 4 + ks * 2 + rb) * 1024);
        const unsigned ew = wait_flag(myws, fl + (unsigned)((j * 4 + c) * 32 + nblk) * 128u, call, tmo, ok);
        const char* wbp = data + (size_t)((j * 4 + c) * 32 + nblk) * 8192 + lane * 16;
#pragma unroll
        for (int ks = 0; ks < 2; ++ks)
#pragma unroll
            for (int i = 0; i < 4; ++i) bf[ks][i] = *(const half8*)(wbp + (ks * 4 + i) * 1024);
        const float s = __builtin_ldexpf(1.0f, (int)ex[c] + (int)ew - 254);
#pragma unroll
        for (int rb = 0; rb < 2; ++rb)
#pragma unroll
            for (int i = 0; i < 4; ++i) {
                f32x4 tmp = (f32x4){0.f, 0.f, 0.f, 0.f};
                tmp = __builtin_amdgcn_mfma_f32_16x16x32_f16(a[0][rb], bf[0][i], tmp, 0, 0, 0);
                tmp = __builtin_amdgcn_mfma_f32_16x16x32_f16(a[1][rb], bf[1][i], tmp, 0, 0, 0);
                acc[rb][i] += tmp * s;
            }
    }

#pragma unroll
    for (int rb = 0; rb < 2; ++rb)
#pragma unroll
        for (int r = 0; r < 4; ++r) {
            f32x4 v = (f32x4){acc[rb][0][r], acc[rb][1][r], acc[rb][2][r], acc[rb][3][r]};
            *(f32x4*)&red[j][rb * 16 + lg * 4 + r][lc * 4] = v;
        }
    __syncthreads();
    if (!ok && lane == 0) s_bad = 1;
    __syncthreads();
    {
        const int tt = threadIdx.x;
        const int row = tt >> 4, c4 = tt & 15;
        f32x4 sum = *(const f32x4*)&red[0][row][c4 * 4];
#pragma unroll
        for (int q = 1; q < 8; ++q) sum += *(const f32x4*)&red[q][row][c4 * 4];
        const unsigned tv = __hip_atomic_load(tmo, __ATOMIC_RELAXED, __HIP_MEMORY_SCOPE_AGENT);
        if (s_bad || tv == call) {
            const float qn = __builtin_nanf("");
            sum = (f32x4){qn, qn, qn, qn};
        }
        *(f32x4*)(P.out + (size_t)(mblk * 32 + row) * ND + nblk * 64 + c4 * 4) = sum;
        if (tt == 0) __hip_atomic_store(ctr, call, __ATOMIC_RELAXED, __HIP_MEMORY_SCOPE_AGENT);
    }
}

}

extern "C" void kernel_launch(int rank, int nranks, ncclComm_t comm,
                              hipStream_t stream,
                              void* const* d_in, const int* in_sizes, int n_in,
                              void* d_out, int out_size,
                              const int* in_dtypes, int out_dtype,
                              void* const* d_out_all,
                              void* const* d_ws_all, size_t ws_bytes) {
    if (nranks != NR || (unsigned)rank >= (unsigned)NR || ws_bytes < (size_t)WS_NEED || n_in < 2 ||
        in_sizes[0] != 2048 * KS || in_sizes[1] != KS * ND || out_size != MS * ND) {
        fprintf(stderr, "kernel_launch: unexpected problem geometry\n");
        return;
    }
    Params P;
    P.x = (const float*)d_in[0];
    P.w = (const float*)d_in[1];
    P.out = (float*)d_out;
    for (int p = 0; p < NR; ++p) P.ws[p] = (char*)d_ws_all[p];
    P.rank = rank;
    hipLaunchKernelGGL(gemm_ag_kernel, dim3(NWG), dim3(NTHREADS), 0, stream, P);
    HIP_CHECK(hipGetLastError());
}
```

```cpp
#include <hip/hip_runtime.h>
#include <rccl/rccl.h>
#include <cstdio>
#include <cstdint>
#include <utility>

#define HIP_CHECK(expr) do { hipError_t _e = (expr); if (_e != hipSuccess) { \
    fprintf(stderr, "HIP error %s at %s:%d\n", hipGetErrorString(_e), __FILE__, __LINE__); \
    return; } } while (0)

#define ST_AUX 17
#define THR 3

#define TS_DECL
#define TS(slot) do { } while (0)
#define TS_FLUSH(w) do { } while (0)

namespace {

typedef _Float16 half8 __attribute__((ext_vector_type(8)));
typedef float f32x4 __attribute__((ext_vector_type(4)));
typedef unsigned u32x4 __attribute__((ext_vector_type(4)));
typedef unsigned long long u64;

constexpr int NR = 8;
constexpr int ND = 2048;
constexpr int MS = 256;
constexpr int KS = 256;
constexpr int NWG = 256;
constexpr int NTHREADS = 512;

constexpr unsigned OFF_CTR = 0;
constexpr unsigned OFF_TMO = 65536;
constexpr unsigned OFF_FLAGS = 131072;
constexpr unsigned FW_BYTES = 8 * 4 * 32 * 128;
constexpr unsigned FX_BYTES = 8 * 8 * 4 * 128;
constexpr unsigned FL_BYTES = FW_BYTES + FX_BYTES;
constexpr unsigned OFF_DATA = 1u << 20;
constexpr unsigned WF_BYTES = 8u << 20;
constexpr unsigned XF_BYTES = 1u << 20;
constexpr unsigned DATA_BYTES = WF_BYTES + XF_BYTES;
constexpr unsigned WS_NEED = OFF_DATA + 2 * DATA_BYTES;
constexpr u64 DEADLINE_TICKS = 50000000ull;

struct Params {
    const float* x;
    const float* w;
    float* out;
    char* ws[NR];
    int rank;
};

template <int N> struct IC { static constexpr int value = N; };
template <class F, int... I> __device__ __forceinline__ void sfor_impl(F&& f, std::integer_sequence<int, I...>) { (f(IC<I>{}), ...); }
template <int N, class F> __device__ __forceinline__ void sfor(F&& f) { sfor_impl(f, std::make_integer_sequence<int, N>{}); }
template <int N> __device__ __forceinline__ void wait_vm() { asm volatile("s_waitcnt vmcnt(%0)" ::"n"(N) : "memory"); }

__device__ __forceinline__ unsigned wave_umax(unsigned v) {
#pragma unroll
    for (int o = 32; o; o >>= 1) {
        unsigned t = (unsigned)__shfl_xor((int)v, o);
        v = v > t ? v : t;
    }
    return v;
}
__device__ __forceinline__ unsigned absbits(float v) { return __builtin_bit_cast(unsigned, v) & 0x7fffffffu; }
__device__ __forceinline__ unsigned umax4(unsigned m, f32x4 v) {
    unsigned a = absbits(v[0]), b = absbits(v[1]), c = absbits(v[2]), d = absbits(v[3]);
    a = a > b ? a : b; c = c > d ? c : d; a = a > c ? a : c;
    return m > a ? m : a;
}
__device__ __forceinline__ unsigned block_exp(unsigned lane_max) {
    unsigned e = __builtin_amdgcn_readfirstlane(wave_umax(lane_max)) >> 23;
    e = e < 1u ? 1u : e;
    e = e > 253u ? 253u : e;
    return e;
}
__device__ __forceinline__ void flag_store(char* base, unsigned off, unsigned call, unsigned e) {
    __hip_atomic_store((u64*)(base + off), ((u64)call << 32) | (u64)e, __ATOMIC_RELAXED, __HIP_MEMORY_SCOPE_SYSTEM);
}
__device__ __forceinline__ unsigned wait_flags(const char* base, unsigned off, unsigned call, unsigned* tmo, bool& ok) {
    const u64* f = (const u64*)(base + off);
    u64 v = __hip_atomic_load(f, __ATOMIC_RELAXED, __HIP_MEMORY_SCOPE_SYSTEM);
    if (!__all((unsigned)(v >> 32) >= call)) {
        const u64 t0 = __builtin_amdgcn_s_memrealtime();
        for (;;) {
            __builtin_amdgcn_s_sleep(2);
            v = __hip_atomic_load(f, __ATOMIC_RELAXED, __HIP_MEMORY_SCOPE_SYSTEM);
            if (__all((unsigned)(v >> 32) >= call)) break;
            unsigned t = __builtin_amdgcn_readfirstlane(__hip_atomic_load(tmo, __ATOMIC_RELAXED, __HIP_MEMORY_SCOPE_AGENT));
            if (t == call || __builtin_amdgcn_s_memrealtime() - t0 > DEADLINE_TICKS) {
                __hip_atomic_store(tmo, call, __ATOMIC_RELAXED, __HIP_MEMORY_SCOPE_AGENT);
                ok = false;
                break;
            }
        }
    }
    asm volatile("" ::: "memory");
    return (unsigned)v;
}
__device__ __forceinline__ u32x4 pack8(float a0, float a1, float a2, float a3, float a4, float a5, float a6, float a7, float m) {
    half8 h;
    h[0] = (_Float16)(a0 * m); h[1] = (_Float16)(a1 * m); h[2] = (_Float16)(a2 * m); h[3] = (_Float16)(a3 * m);
    h[4] = (_Float16)(a4 * m); h[5] = (_Float16)(a5 * m); h[6] = (_Float16)(a6 * m); h[7] = (_Float16)(a7 * m);
    return __builtin_bit_cast(u32x4, h);
}

constexpr int NST = 36;
constexpr bool stage_end(int s) { return s == 3 || s == 11 || s == 19 || s == 27 || s == 35; }
constexpr bool loads_after(int s) { return s == 3 || s == 11 || s == 19; }
constexpr int ops_before_store(int s) {
    return (s - THR >= 0 && stage_end(s - THR)) ? 1 : 0;
}
constexpr int pos_of(int s) {
    int n = 0;
    for (int q = 0; q <= s; ++q) {
        n += ops_before_store(q);
        if (q == s) return n;
        n += 1;
        if (loads_after(q)) n += 16;
    }
    return n;
}

__global__ void __launch_bounds__(NTHREADS) gemm_ag_kernel(Params P) {
    __shared__ __attribute__((aligned(16))) float red[8][64][64];
    __shared__ int s_bad;

    const int g = blockIdx.x;
    const int wave = __builtin_amdgcn_readfirstlane(threadIdx.x >> 6);
    const int lane = threadIdx.x & 63;
    const int lg = lane >> 4, lc = lane & 15;
    const int rank = P.rank;
    char* const myws = P.ws[rank];
    const int xcd = g & 7, t = g >> 4;
    TS_DECL;
    TS(0);

    if ((g >> 3) & 1) {
        if (wave >= 2) return;
        const int s_ = 2 * t + wave;
        const int p = s_ & 7;
        const int b = xcd * 4 + (s_ >> 3);
        unsigned* ctr = (unsigned*)(myws + OFF_CTR + (g * 2 + wave) * 128);
        const unsigned call = __builtin_amdgcn_readfirstlane(__hip_atomic_load(ctr, __ATOMIC_RELAXED, __HIP_MEMORY_SCOPE_AGENT)) + 1u;

        const int mblk = b & 7, cx = b >> 3;
        f32x4 xv[2][2][2];
#pragma unroll
        for (int ks = 0; ks < 2; ++ks)
#pragma unroll
            for (int rb = 0; rb < 2; ++rb) {
                const float* src = P.x + (size_t)(p * MS + mblk * 32 + rb * 16 + lc) * KS + cx * 64 + ks * 32 + lg * 8;
                xv[ks][rb][0] = *(const f32x4*)src;
                xv[ks][rb][1] = *(const f32x4*)(src + 4);
            }
        const float* wsrc = P.w + (size_t)(lg * 8) * ND + b * 64 + lc * 4;
        f32x4 wv[2][2][8];
        auto load_w = [&](auto C) {
            constexpr int c = decltype(C)::value;
#pragma unroll
            for (int ks = 0; ks < 2; ++ks)
#pragma unroll
                for (int j = 0; j < 8; ++j) wv[c & 1][ks][j] = *(const f32x4*)(wsrc + (size_t)(c * 64 + ks * 32 + j) * ND);
        };
        load_w(IC<0>{});

        char* const dws = P.ws[p];
        const unsigned par = call & 1u;
        const __amdgpu_buffer_rsrc_t rs = __builtin_amdgcn_make_buffer_rsrc(dws + OFF_DATA + par * DATA_BYTES, 0, (int)DATA_BYTES, 0x00020000);
        const unsigned fl = OFF_FLAGS + par * FL_BYTES;
        const unsigned xoff = WF_BYTES + (unsigned)(((rank * 8 + mblk) * 4 + cx) * 4) * 1024u + lane * 16u;
        const unsigned woff = (unsigned)((rank * 4) * 32 + b) * 8192u + lane * 16u;

        unsigned e[5];
        float mul = 0.f;
        auto flag_of = [&](auto E) {
            constexpr int en = decltype(E)::value;
            constexpr int st = en == 3 ? 0 : (en - 3) / 8;
            const unsigned off = st == 0 ? fl + FW_BYTES + (unsigned)((rank * 8 + mblk) * 4 + cx) * 128u
                                         : fl + (unsigned)((rank * 4 + (st - 1)) * 32 + b) * 128u;
            if (lane == 0) flag_store(dws, off, call, e[st]);
            TS(2 + st);
        };
        sfor<NST>([&](auto S) {
            constexpr int s = decltype(S)::value;
            if constexpr (s == 0) {
                unsigned m = 0;
#pragma unroll
                for (int ks = 0; ks < 2; ++ks)
#pragma unroll
                    for (int rb = 0; rb < 2; ++rb) { m = umax4(m, xv[ks][rb][0]); m = umax4(m, xv[ks][rb][1]); }
                e[0] = block_exp(m);
                mul = __builtin_bit_cast(float, (254u - e[0]) << 23);
            } else if constexpr (s >= 4 && (s - 4) % 8 == 0) {
                constexpr int c = (s - 4) / 8;
                unsigned m = 0;
#pragma unroll
                for (int ks = 0; ks < 2; ++ks)
#pragma unroll
                    for (int j = 0; j < 8; ++j) m = umax4(m, wv[c & 1][ks][j]);
                e[c + 1] = block_exp(m);
                mul = __builtin_bit_cast(float, (254u - e[c + 1]) << 23);
            }
            u32x4 v;
            unsigned off;
            if constexpr (s < 4) {
                const f32x4 a = xv[s >> 1][s & 1][0], c = xv[s >> 1][s & 1][1];
                v = pack8(a[0], a[1], a[2], a[3], c[0], c[1], c[2], c[3], mul);
                off = xoff + s * 1024u;
            } else {
                constexpr int c = (s - 4) / 8, ks = ((s - 4) >> 2) & 1, i = (s - 4) & 3;
                v = pack8(wv[c & 1][ks][0][i], wv[c & 1][ks][1][i], wv[c & 1][ks][2][i], wv[c & 1][ks][3][i],
                          wv[c & 1][ks][4][i], wv[c & 1][ks][5][i], wv[c & 1][ks][6][i], wv[c & 1][ks][7][i], mul);
                off = woff + c * (32u * 8192u) + (ks * 4 + i) * 1024u;
            }
            if constexpr (s - THR >= 0) {
                wait_vm<pos_of(s) - ops_before_store(s) - pos_of(s - THR) - 1>();
                if constexpr (stage_end(s - THR)) flag_of(IC<s - THR>{});
            }
            __builtin_amdgcn_raw_buffer_store_b128(v, rs, off, 0, ST_AUX);
            asm volatile("" ::: "memory");
            if constexpr (s == 3) TS(1);
            if constexpr (loads_after(s)) { load_w(IC<(s - 3) / 8 + 1>{}); asm volatile("" ::: "memory"); }
        });
        sfor<NST>([&](auto S) {
            constexpr int s = decltype(S)::value;
            if constexpr (stage_end(s) && s + THR >= NST) {
                wait_vm<NST - 1 - s>();
                flag_of(IC<s>{});
            }
        });
        if (lane == 0) __hip_atomic_store(ctr, call, __ATOMIC_RELAXED, __HIP_MEMORY_SCOPE_AGENT);
        TS_FLUSH(8 + wave);
        return;
    }

    const int j = wave;
    const int nblk = xcd * 4 + (t & 3);
    const int m64 = t >> 2;
    unsigned* ctr = (unsigned*)(myws + OFF_CTR + (g * 2) * 128);
    unsigned* tmo = (unsigned*)(myws + OFF_TMO);
    const unsigned call = __builtin_amdgcn_readfirstlane(__hip_atomic_load(ctr, __ATOMIC_RELAXED, __HIP_MEMORY_SCOPE_AGENT)) + 1u;
    const unsigned par = call & 1u;
    const unsigned fl = OFF_FLAGS + par * FL_BYTES;
    const char* data = myws + OFF_DATA + par * DATA_BYTES;
    if (threadIdx.x == 0) s_bad = 0;

    bool ok = true;
    const unsigned exv = wait_flags(myws, fl + FW_BYTES + (unsigned)((j * 8 + m64 * 2) * 4 + (lane & 7)) * 128u, call, tmo, ok);
    TS(1);
    f32x4 acc[4][4];
#pragma unroll
    for (int rb = 0; rb < 4; ++rb)
#pragma unroll
        for (int i = 0; i < 4; ++i) acc[rb][i] = (f32x4){0.f, 0.f, 0.f, 0.f};

    const char* xa = data + WF_BYTES + (size_t)((j * 8 + m64 * 2) * 16) * 1024 + lane * 16;
#pragma unroll
    for (int c = 0; c < 4; ++c) {
        half8 a[2][4], bf[2][4];
#pragma unroll
        for (int h = 0; h < 2; ++h)
#pragma unroll
            for (int ks = 0; ks < 2; ++ks)
#pragma unroll
                for (int rb = 0; rb < 2; ++rb) a[ks][h * 2 + rb] = *(const half8*)(xa + (h * 16 + c * 4 + ks * 2 + rb) * 1024);
        const unsigned ew = __builtin_amdgcn_readfirstlane(wait_flags(myws, fl + (unsigned)((j * 4 + c) * 32 + nblk) * 128u, call, tmo, ok));
        TS(2 + c);
        const char* wbp = data + (size_t)((j * 4 + c) * 32 + nblk) * 8192 + lane * 16;
#pragma unroll
        for (int ks = 0; ks < 2; ++ks)
#pragma unroll
            for (int i = 0; i < 4; ++i) bf[ks][i] = *(const half8*)(wbp + (ks * 4 + i) * 1024);
#pragma unroll
        for (int rb = 0; rb < 4; ++rb) {
            const unsigned ex = __builtin_amdgcn_readlane(exv, (rb >> 1) * 4 + c);
            const float s = __builtin_ldexpf(1.0f, (int)ex + (int)ew - 254);
#pragma unroll
            for (int i = 0; i < 4; ++i) {
                f32x4 tmp = (f32x4){0.f, 0.f, 0.f, 0.f};
                tmp = __builtin_amdgcn_mfma_f32_16x16x32_f16(a[0][rb], bf[0][i], tmp, 0, 0, 0);
                tmp = __builtin_amdgcn_mfma_f32_16x16x32_f16(a[1][rb], bf[1][i], tmp, 0, 0, 0);
                acc[rb][i] += tmp * s;
            }
        }
    }

#pragma unroll
    for (int rb = 0; rb < 4; ++rb)
#pragma unroll
        for (int r = 0; r < 4; ++r) {
            f32x4 v = (f32x4){acc[rb][0][r], acc[rb][1][r], acc[rb][2][r], acc[rb][3][r]};
            *(f32x4*)&red[j][rb * 16 + lg * 4 + r][lc * 4] = v;
        }
    TS(6);
    __syncthreads();
    TS(7);
    if (!ok && lane == 0) s_bad = 1;
    __syncthreads();
    {
        const unsigned tv = __hip_atomic_load(tmo, __ATOMIC_RELAXED, __HIP_MEMORY_SCOPE_AGENT);
#pragma unroll
        for (int h = 0; h < 2; ++h) {
            const int tt = threadIdx.x + h * 512;
            const int row = tt >> 4, c4 = tt & 15;
            f32x4 sum = *(const f32x4*)&red[0][row][c4 * 4];
#pragma unroll
            for (int q = 1; q < 8; ++q) sum += *(const f32x4*)&red[q][row][c4 * 4];
            if (s_bad || tv == call) {
                const float qn = __builtin_nanf("");
                sum = (f32x4){qn, qn, qn, qn};
            }
            *(f32x4*)(P.out + (size_t)(m64 * 64 + row) * ND + nblk * 64 + c4 * 4) = sum;
        }
        if (threadIdx.x == 0) __hip_atomic_store(ctr, call, __ATOMIC_RELAXED, __HIP_MEMORY_SCOPE_AGENT);
    }
    TS(8);
    TS_FLUSH(wave);
}

}

extern "C" void kernel_launch(int rank, int nranks, ncclComm_t comm,
                              hipStream_t stream,
                              void* const* d_in, const int* in_sizes, int n_in,
                              void* d_out, int out_size,
                              const int* in_dtypes, int out_dtype,
                              void* const* d_out_all,
                              void* const* d_ws_all, size_t ws_bytes) {
    if (nranks != NR || (unsigned)rank >= (unsigned)NR || ws_bytes < (size_t)WS_NEED || n_in < 2 ||
        in_sizes[0] != 2048 * KS || in_sizes[1] != KS * ND || out_size != MS * ND) {
        fprintf(stderr, "kernel_launch: unexpected problem geometry\n");
        return;
    }
    Params P;
    P.x = (const float*)d_in[0];
    P.w = (const float*)d_in[1];
    P.out = (float*)d_out;
    for (int p = 0; p < NR; ++p) P.ws[p] = (char*)d_ws_all[p];
    P.rank = rank;
    hipLaunchKernelGGL(gemm_ag_kernel, dim3(NWG), dim3(NTHREADS), 0, stream, P);
    HIP_CHECK(hipGetLastError());
}
```

```cpp
#include <hip/hip_runtime.h>
#include <rccl/rccl.h>
#include <cstdio>
#include <cstdint>
#include <utility>

#define HIP_CHECK(expr) do { hipError_t _e = (expr); if (_e != hipSuccess) { \
    fprintf(stderr, "HIP error %s at %s:%d\n", hipGetErrorString(_e), __FILE__, __LINE__); \
    return; } } while (0)

#define ST_AUX 17
#define THR 5
#define THRB 8
#define ZERO_AUX 17
#define ZTHR 2
#define POLL_SLEEP 4
#define HOT_SLEEP 1

#define TS_DECL
#define TS(slot) do { } while (0)
#define TS_FLUSH(w) do { } while (0)

namespace {

typedef _Float16 half8 __attribute__((ext_vector_type(8)));
typedef float f32x2 __attribute__((ext_vector_type(2)));
typedef float f32x4 __attribute__((ext_vector_type(4)));
typedef unsigned u32x4 __attribute__((ext_vector_type(4)));
typedef unsigned long long u64;

constexpr int NR = 8;
constexpr int ND = 2048;
constexpr int MS = 256;
constexpr int KS = 256;
constexpr int NPROD = 32;
constexpr int NFOR = 64;
constexpr int NOWN = 128;
constexpr int NJAN = 16;
constexpr int NWG = NPROD + NFOR + NOWN + NJAN;
constexpr int NTHREADS = 512;

constexpr unsigned OFF_CTR = 0;
constexpr unsigned OFF_TMO = 65536;
constexpr unsigned OFF_DATA = 1u << 20;
constexpr unsigned XF_OFF = 0;
constexpr unsigned WF_OFF = 2u << 20;
constexpr unsigned RB_OFF = 6u << 20;
constexpr unsigned ZERO_BYTES = 7u << 20;
constexpr unsigned FX_OFF = 7u << 20;
constexpr unsigned FW_OFF = FX_OFF + 4096;
constexpr unsigned BUF_BYTES = (7u << 20) + 65536;
constexpr unsigned WS_NEED = OFF_DATA + 3 * BUF_BYTES;
constexpr u64 DEADLINE_TICKS = 50000000ull;

struct Params {
    const float* x;
    const float* w;
    float* out;
    char* myws;
    char* ws[NR];
    int rank;
};

template <int N> __device__ __forceinline__ void wait_vm() { asm volatile("s_waitcnt vmcnt(%0)" ::"n"(N) : "memory"); }

template <int CTRL> __device__ __forceinline__ unsigned dpp_umax(unsigned v) {
    const unsigned t = (unsigned)__builtin_amdgcn_update_dpp(0, (int)v, CTRL, 0xf, 0xf, false);
    return v > t ? v : t;
}
__device__ __forceinline__ unsigned wave_umax(unsigned v) {
    v = dpp_umax<0xB1>(v);
    v = dpp_umax<0x4E>(v);
    v = dpp_umax<0x141>(v);
    v = dpp_umax<0x140>(v);
    unsigned a = __builtin_amdgcn_readlane(v, 0), b = __builtin_amdgcn_readlane(v, 16);
    unsigned c = __builtin_amdgcn_readlane(v, 32), d = __builtin_amdgcn_readlane(v, 48);
    a = a > b ? a : b; c = c > d ? c : d;
    return a > c ? a : c;
}
__device__ __forceinline__ unsigned absbits(float v) { return __builtin_bit_cast(unsigned, v) & 0x7fffffffu; }
__device__ __forceinline__ unsigned umax4(unsigned m, f32x4 v) {
    unsigned a = absbits(v[0]), b = absbits(v[1]), c = absbits(v[2]), d = absbits(v[3]);
    a = a > b ? a : b; c = c > d ? c : d; a = a > c ? a : c;
    return m > a ? m : a;
}
__device__ __forceinline__ unsigned clamp_exp(unsigned m) {
    unsigned e = m >> 23;
    e = e < 1u ? 1u : e;
    return e > 253u ? 253u : e;
}
__device__ __forceinline__ void word_store(char* p, unsigned call, unsigned e) {
    __hip_atomic_store((u64*)p, ((u64)call << 32) | (u64)e, __ATOMIC_RELAXED, __HIP_MEMORY_SCOPE_SYSTEM);
}
__device__ __forceinline__ u32x4 pack8(float a0, float a1, float a2, float a3, float a4, float a5, float a6, float a7, float m) {
    half8 h;
    h[0] = (_Float16)(a0 * m); h[1] = (_Float16)(a1 * m); h[2] = (_Float16)(a2 * m); h[3] = (_Float16)(a3 * m);
    h[4] = (_Float16)(a4 * m); h[5] = (_Float16)(a5 * m); h[6] = (_Float16)(a6 * m); h[7] = (_Float16)(a7 * m);
    u32x4 v = __builtin_bit_cast(u32x4, h);
#pragma unroll
    for (int k = 0; k < 4; ++k) v[k] = v[k] ? v[k] : 0x8000u;
    return v;
}
__device__ __forceinline__ unsigned umin4(unsigned m, u32x4 v) {
    unsigned a = v[0] < v[1] ? v[0] : v[1], b = v[2] < v[3] ? v[2] : v[3];
    a = a < b ? a : b;
    return m < a ? m : a;
}
__device__ __forceinline__ unsigned ctr_issue(const unsigned* p) {
    unsigned v;
    asm volatile("s_load_dword %0, %1, 0x0" : "=s"(v) : "s"(p) : "memory");
    return v;
}
__device__ __forceinline__ unsigned ctr_wait(unsigned v) {
    unsigned r;
    asm volatile("s_waitcnt lgkmcnt(0)\n\ts_add_u32 %0, %1, 1" : "=s"(r) : "s"(v) : "memory", "scc");
    return r;
}

struct Waiter {
    unsigned* tmo;
    unsigned call;
    u64 t0;
    bool ok;
    __device__ __forceinline__ bool expired() {
        if (!ok) return true;
        if (__builtin_amdgcn_s_memrealtime() - t0 > DEADLINE_TICKS) {
            __hip_atomic_store(tmo, call, __ATOMIC_RELAXED, __HIP_MEMORY_SCOPE_AGENT);
            ok = false;
            return true;
        }
        return false;
    }
};

template <int N>
__device__ __forceinline__ void wait_landed(__amdgpu_buffer_rsrc_t rs, unsigned base, int lane, Waiter& W) {
    const unsigned probe = base - lane * 16u + (unsigned)(lane < N ? lane : N - 1) * 1024u + 1020u;
    for (;;) {
        unsigned v = 1u;
        if (lane == N - 1) v = __builtin_amdgcn_raw_buffer_load_b32(rs, probe, 0, 17);
        if (__all(v != 0u)) break;
        if (W.expired()) return;
        __builtin_amdgcn_s_sleep(POLL_SLEEP);
    }
}
template <int N, class OFF>
__device__ __forceinline__ void validate(u32x4* r, __amdgpu_buffer_rsrc_t rs, unsigned base, OFF off, Waiter& W) {
    unsigned m = 0xffffffffu;
#pragma unroll
    for (int f = 0; f < N; ++f) m = umin4(m, r[f]);
    while (__builtin_expect(!__all(m != 0u), 0)) {
        if (W.expired()) break;
#pragma unroll
        for (int f = 0; f < N; ++f) r[f] = __builtin_amdgcn_raw_buffer_load_b128(rs, base + off(f), 0, 17);
        m = 0xffffffffu;
#pragma unroll
        for (int f = 0; f < N; ++f) m = umin4(m, r[f]);
    }
}
struct Lin { __device__ __forceinline__ unsigned operator()(int f) const { return (unsigned)f * 1024u; } };

template <int N>
__device__ __forceinline__ void acquire(u32x4* r, __amdgpu_buffer_rsrc_t rs, unsigned base, int lane, Waiter& W) {
    wait_landed<N>(rs, base, lane, W);
#pragma unroll
    for (int f = 0; f < N; ++f) r[f] = __builtin_amdgcn_raw_buffer_load_b128(rs, base + f * 1024u, 0, 0);
    validate<N>(r, rs, base, Lin{}, W);
}

__device__ __forceinline__ f32x4 mfma(u32x4 a, u32x4 b, f32x4 c) {
    return __builtin_amdgcn_mfma_f32_16x16x32_f16(__builtin_bit_cast(half8, a), __builtin_bit_cast(half8, b), c, 0, 0, 0);
}

__device__ __forceinline__ unsigned wait_words(const char* p, unsigned call, Waiter& W) {
    u64 v;
    for (;;) {
        v = __hip_atomic_load((const u64*)p, __ATOMIC_RELAXED, __HIP_MEMORY_SCOPE_SYSTEM);
        if (__all((unsigned)(v >> 32) == call)) break;
        if (W.expired()) break;
        __builtin_amdgcn_s_sleep(2);
    }
    return (unsigned)v;
}

__global__ void __launch_bounds__(NTHREADS) gemm_ag_kernel(Params P) {
    __shared__ __attribute__((aligned(16))) float red[8 * 2 * 16 * 68];
    __shared__ unsigned s_m[4];
    __shared__ int s_bad;

    const int g = blockIdx.x;
    const int wave = __builtin_amdgcn_readfirstlane(threadIdx.x >> 6);
    const int lane = threadIdx.x & 63;
    const int lg = lane >> 4, lc = lane & 15;
    const int rank = P.rank;
    char* const myws = P.myws;
    unsigned* ctr = (unsigned*)(myws + OFF_CTR + g * 128);
    TS_DECL;
    TS(0);
    const float* const px = P.x;
    const float* const pw = P.w;
    char* const wsx = P.ws[g < NPROD ? ((g >> 1) & 3) * 2 + (g & 1) : (g & 7)];
    asm volatile("" :: "s"(px), "s"(pw), "s"(myws), "s"(wsx), "s"(rank));
    const unsigned c0 = ctr_issue(ctr);

    if (g < NPROD) {
        if (wave >= 4) return;
        const int half = g & 1, blk = g >> 1;
        const int t = ((g >> 1) & 3) * 2 + half;
        const int rb16 = (g >> 3) * 4 + wave;
        const int rowb = MS * t + 16 * rb16 + lc;
        f32x4 xb[8][2], wv[2][8];
        {
            const float* wsrc = pw + (size_t)(wave * 64 + lg * 8) * ND + half * 1024 + blk * 64 + lc * 4;
#pragma unroll
            for (int ks = 0; ks < 2; ++ks)
#pragma unroll
                for (int jj = 0; jj < 8; ++jj) wv[ks][jj] = *(const f32x4*)(wsrc + (size_t)(ks * 32 + jj) * ND);
        }
        asm volatile("" ::: "memory");
#pragma unroll
        for (int s = 0; s < 8; ++s) {
            const float* src = px + (size_t)rowb * KS + s * 32 + lg * 8;
            xb[s][0] = *(const f32x4*)src;
            xb[s][1] = *(const f32x4*)(src + 4);
        }
        const unsigned call = ctr_wait(c0);
        const unsigned boff = OFF_DATA + (call % 3u) * BUF_BYTES;
        char* const dx = wsx + boff;
        const __amdgpu_buffer_rsrc_t rx = __builtin_amdgcn_make_buffer_rsrc(dx, 0, (int)BUF_BYTES, 0x00020000);
        {
            unsigned m = 0;
#pragma unroll
            for (int ks = 0; ks < 2; ++ks)
#pragma unroll
                for (int jj = 0; jj < 8; ++jj) m = umax4(m, wv[ks][jj]);
            m = wave_umax(m);
            if (lane == 0) s_m[wave] = m;
        }
        TS(1);
        asm volatile("s_waitcnt lgkmcnt(0)\n\ts_barrier" ::: "memory");
        unsigned ew;
        {
            unsigned m = s_m[0];
#pragma unroll
            for (int k = 1; k < 4; ++k) m = s_m[k] > m ? s_m[k] : m;
            ew = __builtin_amdgcn_readfirstlane(clamp_exp(m));
        }
        if (lane == 0) word_store(P.ws[wave * 2 + half] + boff + FW_OFF + (rank * 16 + blk) * 8, call, ew);
        __amdgpu_buffer_rsrc_t rw[4];
#pragma unroll
        for (int dd = 0; dd < 4; ++dd) {
            const int d = (((dd + wave + blk) & 3) << 1) + half;
            rw[dd] = __builtin_amdgcn_make_buffer_rsrc(P.ws[d] + boff, 0, (int)BUF_BYTES, 0x00020000);
        }
        unsigned eb = 0;
        TS(2);
        {
            const float mul = __builtin_bit_cast(float, (254u - ew) << 23);
            const unsigned off = WF_OFF + (unsigned)((rank * 16 + blk) * 32 + wave * 2) * 1024u + lane * 16u;
#pragma unroll
            for (int s = 0; s < 4; ++s) {
                u32x4 v[2];
#pragma unroll
                for (int ks = 0; ks < 2; ++ks)
                    v[ks] = pack8(wv[ks][0][s], wv[ks][1][s], wv[ks][2][s], wv[ks][3][s], wv[ks][4][s], wv[ks][5][s], wv[ks][6][s], wv[ks][7][s], mul);
#pragma unroll
                for (int dd = 0; dd < 4; ++dd)
#pragma unroll
                    for (int ks = 0; ks < 2; ++ks) {
                        if (s * 8 + dd * 2 + ks >= THR) wait_vm<THR - 1>();
                        __builtin_amdgcn_raw_buffer_store_b128(v[ks], rw[dd], off + s * 8192u + ks * 1024u, 0, ST_AUX);
                        asm volatile("" ::: "memory");
                    }
                if (s == 0) {
                    unsigned m = 0;
#pragma unroll
                    for (int q = 0; q < 8; ++q) { m = umax4(m, xb[q][0]); m = umax4(m, xb[q][1]); }
                    eb = __builtin_amdgcn_readfirstlane(clamp_exp(wave_umax(m)));
                    if (lane == 0) word_store(dx + FX_OFF + (rank * 32 + 16 + rb16) * 8, call, eb);
                }
                TS(3 + s);
            }
        }
        {
            const float mul = __builtin_bit_cast(float, (254u - eb) << 23);
            const unsigned off = XF_OFF + (unsigned)((rank * 32 + 16 + rb16) * 8) * 1024u + lane * 16u;
#pragma unroll
            for (int s = 0; s < 8; ++s) {
                const u32x4 v = pack8(xb[s][0][0], xb[s][0][1], xb[s][0][2], xb[s][0][3], xb[s][1][0], xb[s][1][1], xb[s][1][2], xb[s][1][3], mul);
                wait_vm<THRB - 1>();
                __builtin_amdgcn_raw_buffer_store_b128(v, rx, off + s * 1024u, 0, ST_AUX);
                asm volatile("" ::: "memory");
            }
        }
        TS(7);
        if (threadIdx.x == 0) __hip_atomic_store(ctr, call, __ATOMIC_RELAXED, __HIP_MEMORY_SCOPE_AGENT);
        TS_FLUSH(8 + wave);
        return;
    }

    if (g >= NPROD + NFOR + NOWN) {
        const unsigned call = ctr_wait(c0);
        const __amdgpu_buffer_rsrc_t rz = __builtin_amdgcn_make_buffer_rsrc(myws + OFF_DATA + ((call + 2u) % 3u) * BUF_BYTES, 0, (int)ZERO_BYTES, 0x00020000);
        const unsigned zo = (unsigned)((g - NPROD - NFOR - NOWN) * 8 + wave) * 57344u + lane * 16u;
#pragma unroll 1
        for (int k = 0; k < 56; ++k) {
            wait_vm<ZTHR - 1>();
            __builtin_amdgcn_raw_buffer_store_b128((u32x4){0u, 0u, 0u, 0u}, rz, zo + k * 1024u, 0, ZERO_AUX);
        }
        if (threadIdx.x == 0) __hip_atomic_store(ctr, call, __ATOMIC_RELAXED, __HIP_MEMORY_SCOPE_AGENT);
        return;
    }

    if (g >= NPROD + NFOR && wave == 0) {
        const int xt = g & 7, xrb = (g - NPROD - NFOR) >> 3;
        const int rowa = MS * (2 * (xrb >> 2) + 1 - (xt & 1)) + 64 * (xt >> 1) + 16 * (xrb & 3) + lc;
        f32x4 xa[8][2];
#pragma unroll
        for (int s = 0; s < 8; ++s) {
            const float* src = px + (size_t)rowa * KS + s * 32 + lg * 8;
            xa[s][0] = *(const f32x4*)src;
            xa[s][1] = *(const f32x4*)(src + 4);
        }
        const unsigned call = ctr_wait(c0);
        char* const dx = wsx + OFF_DATA + (call % 3u) * BUF_BYTES;
        const __amdgpu_buffer_rsrc_t rx = __builtin_amdgcn_make_buffer_rsrc(dx, 0, (int)BUF_BYTES, 0x00020000);
        unsigned m = 0;
#pragma unroll
        for (int s = 0; s < 8; ++s) { m = umax4(m, xa[s][0]); m = umax4(m, xa[s][1]); }
        const unsigned ea = __builtin_amdgcn_readfirstlane(clamp_exp(wave_umax(m)));
        const float mul = __builtin_bit_cast(float, (254u - ea) << 23);
        const unsigned off = XF_OFF + (unsigned)((rank * 32 + xrb) * 8) * 1024u + lane * 16u;
#pragma unroll
        for (int s = 0; s < 8; ++s) {
            const u32x4 v = pack8(xa[s][0][0], xa[s][0][1], xa[s][0][2], xa[s][0][3], xa[s][1][0], xa[s][1][1], xa[s][1][2], xa[s][1][3], mul);
            __builtin_amdgcn_raw_buffer_store_b128(v, rx, off + s * 1024u, 0, ST_AUX);
        }
        if (lane == 0) word_store(dx + FX_OFF + (rank * 32 + xrb) * 8, call, ea);
        TS(9);
    }
    const unsigned call = ctr_wait(c0);
    const unsigned boff = OFF_DATA + (call % 3u) * BUF_BYTES;
    const int j = wave;
    const int h = rank & 1;
    const __amdgpu_buffer_rsrc_t rs = __builtin_amdgcn_make_buffer_rsrc(myws + boff, 0, (int)BUF_BYTES, 0x00020000);
    if (threadIdx.x == 0) s_bad = 0;
    Waiter W;
    W.tmo = (unsigned*)(myws + OFF_TMO);
    W.call = call;
    W.t0 = __builtin_amdgcn_s_memrealtime();
    W.ok = true;
    const float qn = __builtin_nanf("");

    if (g < NPROD + NFOR) {
        const int cg = g - NPROD;
        const int blk = (cg & 7) * 2 + ((cg >> 3) & 1);
        const int rb64 = cg >> 4;
        u32x4 a[32];
        acquire<32>(a, rs, XF_OFF + (unsigned)((j * 32 + rb64 * 4) * 8) * 1024u + lane * 16u, lane, W);
        TS(1);
        const unsigned bbase = WF_OFF + (unsigned)((j * 16 + blk) * 32) * 1024u + lane * 16u;
        const int dst = 2 * rb64 + 1 - h;
        const __amdgpu_buffer_rsrc_t rd = __builtin_amdgcn_make_buffer_rsrc(P.ws[dst] + boff, 0, (int)BUF_BYTES, 0x00020000);
        int sc[4];
#pragma unroll
        for (int s = 0; s < 4; ++s) {
            u32x4 b[8];
            acquire<8>(b, rs, bbase + s * 8192u, lane, W);
            f32x4 acc[4];
#pragma unroll
            for (int rb = 0; rb < 4; ++rb) acc[rb] = (f32x4){0.f, 0.f, 0.f, 0.f};
#pragma unroll
            for (int k = 0; k < 8; ++k)
#pragma unroll
                for (int rb = 0; rb < 4; ++rb) acc[rb] = mfma(a[rb * 8 + k], b[k], acc[rb]);
            if (s == 0) {
                const unsigned wo = lane < 4 ? FX_OFF + (unsigned)(j * 32 + rb64 * 4 + lane) * 8u : FW_OFF + (unsigned)(j * 16 + blk) * 8u;
                const unsigned v = wait_words(myws + boff + wo, call, W);
#pragma unroll
                for (int rb = 0; rb < 4; ++rb) sc[rb] = (int)__builtin_amdgcn_readlane(v, rb) + (int)__builtin_amdgcn_readlane(v, 4) - 254;
            }
            float* const slab = red + ((s & 1) * 8 + j) * (16 * 68);
#pragma unroll
            for (int rb = 0; rb < 4; ++rb) {
                f32x4 v;
#pragma unroll
                for (int r = 0; r < 4; ++r) v[r] = __builtin_ldexpf(acc[rb][r], sc[rb]);
                *(f32x4*)(slab + lc * 68 + rb * 16 + lg * 4) = v;
            }
            if (!W.ok && lane == 0) s_bad = 1;
            TS(2 + s);
            __syncthreads();
            if (wave < 4) {
                const float* src = red + (s & 1) * 8 * (16 * 68) + lc * 68 + (threadIdx.x >> 4) * 4;
                f32x4 sum = *(const f32x4*)src;
#pragma unroll
                for (int k = 1; k < 8; ++k) sum += *(const f32x4*)(src + k * (16 * 68));
                if (s_bad) sum = (f32x4){qn, qn, qn, qn};
                u32x4 v = __builtin_bit_cast(u32x4, sum);
#pragma unroll
                for (int k = 0; k < 4; ++k) v[k] = v[k] ? v[k] : 0x80000000u;
                __builtin_amdgcn_raw_buffer_store_b128(v, rd, RB_OFF + (unsigned)(((rank >> 1) * 16 + blk) * 4 + s) * 4096u + threadIdx.x * 16u, 0, ST_AUX);
            }
        }
        TS(6);
        if (wave < 4) {
            const unsigned off = RB_OFF + (unsigned)((rb64 * 16 + blk) * 4) * 4096u + threadIdx.x * 16u;
            u32x4 v[4];
            for (;;) {
#pragma unroll
                for (int s = 0; s < 4; ++s) v[s] = __builtin_amdgcn_raw_buffer_load_b128(rs, off + s * 4096u, 0, 17);
                if (__all(umin4(umin4(umin4(umin4(0xffffffffu, v[0]), v[1]), v[2]), v[3]) != 0u)) break;
                if (W.expired()) break;
                __builtin_amdgcn_s_sleep(HOT_SLEEP);
            }
            float* o = P.out + (size_t)(rb64 * 64 + (threadIdx.x >> 4) * 4) * ND + (1 - h) * 1024 + blk * 64 + lc * 4;
#pragma unroll
            for (int r = 0; r < 4; ++r) {
                f32x4 q = __builtin_bit_cast(f32x4, (u32x4){v[0][r], v[1][r], v[2][r], v[3][r]});
                if (!W.ok) q = (f32x4){qn, qn, qn, qn};
                *(f32x4*)(o + (size_t)r * ND) = q;
            }
        }
        if (threadIdx.x == 0) __hip_atomic_store(ctr, call, __ATOMIC_RELAXED, __HIP_MEMORY_SCOPE_AGENT);
        TS(7);
        TS_FLUSH(wave);
        return;
    }

    {
        const int cg = g - NPROD - NFOR;
        const int blk = (cg & 7) * 2 + ((cg >> 3) & 1);
        const int rb32 = cg >> 4;
        u32x4 b[32];
        const unsigned bbase = WF_OFF + (unsigned)((j * 16 + blk) * 32) * 1024u + lane * 16u;
#pragma unroll
        for (int s = 0; s < 4; ++s) {
            acquire<8>(b + s * 8, rs, bbase + s * 8192u, lane, W);
            TS(1 + s);
        }
        int sc[2];
        {
            const unsigned wo = lane < 2 ? FX_OFF + (unsigned)(j * 32 + 16 + rb32 * 2 + lane) * 8u : FW_OFF + (unsigned)(j * 16 + blk) * 8u;
            const unsigned v = wait_words(myws + boff + wo, call, W);
#pragma unroll
            for (int rb = 0; rb < 2; ++rb) sc[rb] = (int)__builtin_amdgcn_readlane(v, rb) + (int)__builtin_amdgcn_readlane(v, 2) - 254;
        }
        const unsigned abase = XF_OFF + (unsigned)((j * 32 + 16 + rb32 * 2) * 8) * 1024u + lane * 16u;
        f32x4 acc[2][4];
#pragma unroll
        for (int rb = 0; rb < 2; ++rb)
#pragma unroll
            for (int s = 0; s < 4; ++s) acc[rb][s] = (f32x4){0.f, 0.f, 0.f, 0.f};
        for (;;) {
            unsigned v = 1u;
            if (lane == 63) v = __builtin_amdgcn_raw_buffer_load_b32(rs, abase + 8192u + 12u, 0, 17);
            if (__all(v != 0u)) break;
            if (W.expired()) break;
            __builtin_amdgcn_s_sleep(POLL_SLEEP);
        }
        TS(5);
        u32x4 a[2][8];
        const auto aoff = [](int f) { return (unsigned)((f >> 2) * 8192 + (f & 3) * 1024); };
#pragma unroll
        for (int c = 0; c < 2; ++c)
#pragma unroll
            for (int f = 0; f < 8; ++f) a[c][f] = __builtin_amdgcn_raw_buffer_load_b128(rs, abase + c * 4096u + aoff(f), 0, 17);
#pragma unroll
        for (int c = 0; c < 2; ++c) {
            validate<8>(a[c], rs, abase + c * 4096u, aoff, W);
#pragma unroll
            for (int ks = 0; ks < 4; ++ks)
#pragma unroll
                for (int rb = 0; rb < 2; ++rb)
#pragma unroll
                    for (int s = 0; s < 4; ++s) acc[rb][s] = mfma(a[c][rb * 4 + ks], b[s * 8 + c * 4 + ks], acc[rb][s]);
        }
        TS(6);
#pragma unroll
        for (int rb = 0; rb < 2; ++rb)
#pragma unroll
            for (int r = 0; r < 4; ++r) {
                f32x4 v;
#pragma unroll
                for (int s = 0; s < 4; ++s) v[s] = __builtin_ldexpf(acc[rb][s][r], sc[rb]);
                *(f32x4*)(red + (j * 32 + rb * 16 + lg * 4 + r) * 64 + lc * 4) = v;
            }
        if (!W.ok && lane == 0) s_bad = 1;
        __syncthreads();
        TS(7);
        const float* src = red + threadIdx.x * 4;
        f32x4 sum = *(const f32x4*)src;
#pragma unroll
        for (int k = 1; k < 8; ++k) sum += *(const f32x4*)(src + k * 2048);
        if (s_bad) sum = (f32x4){qn, qn, qn, qn};
        *(f32x4*)(P.out + (size_t)(rb32 * 32 + (threadIdx.x >> 4)) * ND + h * 1024 + blk * 64 + lc * 4) = sum;
        if (threadIdx.x == 0) __hip_atomic_store(ctr, call, __ATOMIC_RELAXED, __HIP_MEMORY_SCOPE_AGENT);
        TS(8);
        TS_FLUSH(wave);
    }
}

}

extern "C" void kernel_launch(int rank, int nranks, ncclComm_t comm,
                              hipStream_t stream,
                              void* const* d_in, const int* in_sizes, int n_in,
                              void* d_out, int out_size,
                              const int* in_dtypes, int out_dtype,
                              void* const* d_out_all,
                              void* const* d_ws_all, size_t ws_bytes) {
    if (nranks != NR || (unsigned)rank >= (unsigned)NR || ws_bytes < (size_t)WS_NEED || n_in < 2 ||
        in_sizes[0] != 2048 * KS || in_sizes[1] != KS * ND || out_size != MS * ND) {
        fprintf(stderr, "kernel_launch: unexpected problem geometry\n");
        return;
    }
    Params P;
    P.x = (const float*)d_in[0];
    P.w = (const float*)d_in[1];
    P.out = (float*)d_out;
    for (int p = 0; p < NR; ++p) P.ws[p] = (char*)d_ws_all[p];
    P.rank = rank;
    P.myws = (char*)d_ws_all[rank];
    hipLaunchKernelGGL(gemm_ag_kernel, dim3(NWG), dim3(NTHREADS), 0, stream, P);
    HIP_CHECK(hipGetLastError());
}
```

```cpp
#include <hip/hip_runtime.h>
#include <rccl/rccl.h>
#include <cstdio>
#include <cstdint>
#include <utility>
#include <type_traits>

#define HIP_CHECK(expr) do { hipError_t _e = (expr); if (_e != hipSuccess) { \
    fprintf(stderr, "HIP error %s at %s:%d\n", hipGetErrorString(_e), __FILE__, __LINE__); \
    return; } } while (0)

#define ST_AUX 17
#define THR 4
#define THRB 8
#define RELAY 1
#define THRS 5
#define THRR THRS
#define ZREP 1
#define TXS 0
#define TXO 0
#define PMAP 0
#define RB 4
#define RQ 0
#define RPOS 1
#define RTHR 2
#define ZERO_AUX 17
#define ZTHR 2
#define POLL_SLEEP 4
#define HOT_SLEEP 1

#define TS_DECL
#define TS(slot) do { } while (0)
#define TS_FLUSH(w) do { } while (0)

namespace {

typedef _Float16 half8 __attribute__((ext_vector_type(8)));
typedef float f32x2 __attribute__((ext_vector_type(2)));
typedef float f32x4 __attribute__((ext_vector_type(4)));
typedef unsigned u32x4 __attribute__((ext_vector_type(4)));
typedef unsigned long long u64;

constexpr int NR = 8;
constexpr int ND = 2048;
constexpr int MS = 256;
constexpr int KS = 256;
constexpr int NPROD = 32;
constexpr int NFOR = 64;
constexpr int NOWN = 128;
constexpr int NJAN = 16;
constexpr int NWG = NPROD + NFOR + NOWN + NJAN;
constexpr int NTHREADS = 512;

constexpr unsigned OFF_CTR = 0;
constexpr unsigned OFF_TMO = 65536;
constexpr unsigned OFF_DATA = 1u << 20;
constexpr unsigned XF_OFF = 0;
constexpr unsigned WF_OFF = 2u << 20;
constexpr unsigned RB_OFF = 6u << 20;
constexpr unsigned ZERO_BYTES = 7u << 20;
constexpr unsigned FX_OFF = 7u << 20;
constexpr unsigned FW_OFF = FX_OFF + 4096;
constexpr unsigned BUF_BYTES = (7u << 20) + 65536;
constexpr unsigned WS_NEED = OFF_DATA + 3 * BUF_BYTES;
constexpr u64 DEADLINE_TICKS = 50000000ull;

struct Params {
    const float* x;
    const float* w;
    float* out;
    char* myws;
    char* ws[NR];
    int rank;
};

template <int N> __device__ __forceinline__ void wait_vm() { asm volatile("s_waitcnt vmcnt(%0)" ::"n"(N) : "memory"); }

template <int CTRL> __device__ __forceinline__ unsigned dpp_umax(unsigned v) {
    const unsigned t = (unsigned)__builtin_amdgcn_update_dpp(0, (int)v, CTRL, 0xf, 0xf, false);
    return v > t ? v : t;
}
__device__ __forceinline__ unsigned wave_umax(unsigned v) {
    v = dpp_umax<0xB1>(v);
    v = dpp_umax<0x4E>(v);
    v = dpp_umax<0x141>(v);
    v = dpp_umax<0x140>(v);
    unsigned a = __builtin_amdgcn_readlane(v, 0), b = __builtin_amdgcn_readlane(v, 16);
    unsigned c = __builtin_amdgcn_readlane(v, 32), d = __builtin_amdgcn_readlane(v, 48);
    a = a > b ? a : b; c = c > d ? c : d;
    return a > c ? a : c;
}
__device__ __forceinline__ unsigned absbits(float v) { return __builtin_bit_cast(unsigned, v) & 0x7fffffffu; }
__device__ __forceinline__ unsigned umax4(unsigned m, f32x4 v) {
    unsigned a = absbits(v[0]), b = absbits(v[1]), c = absbits(v[2]), d = absbits(v[3]);
    a = a > b ? a : b; c = c > d ? c : d; a = a > c ? a : c;
    return m > a ? m : a;
}
__device__ __forceinline__ unsigned clamp_exp(unsigned m) {
    unsigned e = m >> 23;
    e = e < 1u ? 1u : e;
    return e > 253u ? 253u : e;
}
__device__ __forceinline__ void word_store(char* p, unsigned call, unsigned e) {
    __hip_atomic_store((u64*)p, ((u64)call << 32) | (u64)e, __ATOMIC_RELAXED, __HIP_MEMORY_SCOPE_SYSTEM);
}
__device__ __forceinline__ u32x4 pack8(float a0, float a1, float a2, float a3, float a4, float a5, float a6, float a7, float m) {
    half8 h;
    h[0] = (_Float16)(a0 * m); h[1] = (_Float16)(a1 * m); h[2] = (_Float16)(a2 * m); h[3] = (_Float16)(a3 * m);
    h[4] = (_Float16)(a4 * m); h[5] = (_Float16)(a5 * m); h[6] = (_Float16)(a6 * m); h[7] = (_Float16)(a7 * m);
    u32x4 v = __builtin_bit_cast(u32x4, h);
#pragma unroll
    for (int k = 0; k < 4; ++k) v[k] = v[k] ? v[k] : 0x8000u;
    return v;
}
__device__ __forceinline__ unsigned umin4(unsigned m, u32x4 v) {
    unsigned a = v[0] < v[1] ? v[0] : v[1], b = v[2] < v[3] ? v[2] : v[3];
    a = a < b ? a : b;
    return m < a ? m : a;
}
__device__ __forceinline__ unsigned ctr_issue(const unsigned* p) {
    unsigned v;
    asm volatile("s_load_dword %0, %1, 0x0" : "=s"(v) : "s"(p) : "memory");
    return v;
}
__device__ __forceinline__ unsigned ctr_wait(unsigned v) {
    unsigned r;
    asm volatile("s_waitcnt lgkmcnt(0)\n\ts_add_u32 %0, %1, 1" : "=s"(r) : "s"(v) : "memory", "scc");
    return r;
}

struct Waiter {
    unsigned* tmo;
    unsigned call;
    u64 t0;
    bool ok;
    __device__ __forceinline__ bool expired() {
        if (!ok) return true;
        if (__builtin_amdgcn_s_memrealtime() - t0 > DEADLINE_TICKS) {
            __hip_atomic_store(tmo, call, __ATOMIC_RELAXED, __HIP_MEMORY_SCOPE_AGENT);
            ok = false;
            return true;
        }
        return false;
    }
};

template <int N>
__device__ __forceinline__ void wait_landed(__amdgpu_buffer_rsrc_t rs, unsigned base, int lane, Waiter& W) {
    const unsigned probe = base - lane * 16u + (unsigned)(lane < N ? lane : N - 1) * 1024u + 1020u;
    for (;;) {
        unsigned v = 1u;
        if (lane == N - 1) v = __builtin_amdgcn_raw_buffer_load_b32(rs, probe, 0, 17);
        if (__all(v != 0u)) break;
        if (W.expired()) return;
        __builtin_amdgcn_s_sleep(POLL_SLEEP);
    }
}
template <int N, class OFF>
__device__ __forceinline__ void validate(u32x4* r, __amdgpu_buffer_rsrc_t rs, unsigned base, OFF off, Waiter& W) {
    unsigned m = 0xffffffffu;
#pragma unroll
    for (int f = 0; f < N; ++f) m = umin4(m, r[f]);
    while (__builtin_expect(!__all(m != 0u), 0)) {
        if (W.expired()) break;
#pragma unroll
        for (int f = 0; f < N; ++f) r[f] = __builtin_amdgcn_raw_buffer_load_b128(rs, base + off(f), 0, 17);
        m = 0xffffffffu;
#pragma unroll
        for (int f = 0; f < N; ++f) m = umin4(m, r[f]);
    }
}
struct Lin { __device__ __forceinline__ unsigned operator()(int f) const { return (unsigned)f * 1024u; } };

template <int N>
__device__ __forceinline__ void acquire(u32x4* r, __amdgpu_buffer_rsrc_t rs, unsigned base, int lane, Waiter& W) {
    wait_landed<N>(rs, base, lane, W);
#pragma unroll
    for (int f = 0; f < N; ++f) r[f] = __builtin_amdgcn_raw_buffer_load_b128(rs, base + f * 1024u, 0, 0);
    validate<N>(r, rs, base, Lin{}, W);
}

#define OUT_AUX 16
__device__ __forceinline__ f32x4 mfma(u32x4 a, u32x4 b, f32x4 c) {
    return __builtin_amdgcn_mfma_f32_16x16x32_f16(__builtin_bit_cast(half8, a), __builtin_bit_cast(half8, b), c, 0, 0, 0);
}

__device__ __forceinline__ unsigned wait_words(const char* p, unsigned call, Waiter& W) {
    u64 v;
    for (;;) {
        v = __hip_atomic_load((const u64*)p, __ATOMIC_RELAXED, __HIP_MEMORY_SCOPE_SYSTEM);
        if (__all((unsigned)(v >> 32) == call)) break;
        if (W.expired()) break;
        __builtin_amdgcn_s_sleep(2);
    }
    return (unsigned)v;
}

__global__ void __launch_bounds__(NTHREADS) gemm_ag_kernel(Params P) {
    __shared__ __attribute__((aligned(16))) float red[4 * 8 * 16 * 68];
    __shared__ unsigned s_m[4];
    __shared__ unsigned s_cnt[4];
    __shared__ int s_bad;

    const int g = blockIdx.x;
    const int wave = __builtin_amdgcn_readfirstlane(threadIdx.x >> 6);
    const int lane = threadIdx.x & 63;
    const int lg = lane >> 4, lc = lane & 15;
    const int rank = P.rank;
    char* const myws = P.myws;
    unsigned* ctr = (unsigned*)(myws + OFF_CTR + g * 128);
    TS_DECL;
    TS(0);
    const float* const px = P.x;
    const float* const pw = P.w;
    char* const wsx = P.ws[g < NPROD ? ((g >> 1) & 3) * 2 + (g & 1) : (g & 7)];
    asm volatile("" :: "s"(px), "s"(pw), "s"(myws), "s"(wsx), "s"(rank));
    const unsigned c0 = ctr_issue(ctr);
    const u64 tstart = __builtin_amdgcn_s_memrealtime();

    if (g < NPROD) {
        if (wave >= 4) return;
        const int half = g & 1, blk = g >> 1;
        const int t = (blk & 3) * 2 + half;
        const int rb16 = (blk >> 2) * 4 + wave;
        const int rowb = MS * t + 16 * rb16 + lc;
        f32x4 xb[8][2], wv[2][8];
        {
            const float* wsrc = pw + (size_t)(wave * 64 + lg * 8) * ND + half * 1024 + blk * 64 + lc * 4;
#pragma unroll
            for (int ks = 0; ks < 2; ++ks)
#pragma unroll
                for (int jj = 0; jj < 8; ++jj) wv[ks][jj] = *(const f32x4*)(wsrc + (size_t)(ks * 32 + jj) * ND);
        }
        asm volatile("" ::: "memory");
#pragma unroll
        for (int s = 0; s < 8; ++s) {
            const float* src = px + (size_t)rowb * KS + s * 32 + lg * 8;
            xb[s][0] = *(const f32x4*)src;
            xb[s][1] = *(const f32x4*)(src + 4);
        }
        const unsigned call = ctr_wait(c0);
        const unsigned boff = OFF_DATA + (call % 3u) * BUF_BYTES;
        char* const dx = wsx + boff;
        const __amdgpu_buffer_rsrc_t rx = __builtin_amdgcn_make_buffer_rsrc(dx, 0, (int)BUF_BYTES, 0x00020000);
        {
            unsigned m = 0;
#pragma unroll
            for (int ks = 0; ks < 2; ++ks)
#pragma unroll
                for (int jj = 0; jj < 8; ++jj) m = umax4(m, wv[ks][jj]);
            m = wave_umax(m);
            if (lane == 0) s_m[wave] = m;
        }
        TS(1);
        asm volatile("s_waitcnt lgkmcnt(0)\n\ts_barrier" ::: "memory");
        unsigned ew;
        {
            unsigned m = s_m[0];
#pragma unroll
            for (int k = 1; k < 4; ++k) m = s_m[k] > m ? s_m[k] : m;
            ew = __builtin_amdgcn_readfirstlane(clamp_exp(m));
        }
        if (lane == 0) word_store(P.ws[wave * 2 + half] + boff + FW_OFF + (rank * 16 + blk) * 8, call, ew);
        __amdgpu_buffer_rsrc_t rw[4];
#pragma unroll
        for (int dd = 0; dd < 4; ++dd) {
            const int d = (((dd + wave + blk) & 3) << 1) + half;
            rw[dd] = __builtin_amdgcn_make_buffer_rsrc(P.ws[d] + boff, 0, (int)BUF_BYTES, 0x00020000);
        }
        unsigned eb = 0;
        TS(2);
        {
            const float mul = __builtin_bit_cast(float, (254u - ew) << 23);
            const unsigned off = WF_OFF + (unsigned)((rank * 16 + blk) * 32 + wave * 2) * 1024u + lane * 16u;
            const __amdgpu_buffer_rsrc_t rr = __builtin_amdgcn_make_buffer_rsrc(P.ws[((blk & 3) << 1) + half] + boff, 0, (int)BUF_BYTES, 0x00020000);
            const __amdgpu_buffer_rsrc_t rme = __builtin_amdgcn_make_buffer_rsrc(myws + boff, 0, (int)BUF_BYTES, 0x00020000);
            const unsigned roff = WF_OFF + (unsigned)(((2 * (blk & 3) + 1 - half) * 16 + (blk & 12) + (rank >> 1)) * 32 + 24 + wave * 2) * 1024u + lane * 16u;
            u32x4 rl[2];
            const auto push = [&](auto relayed) {
                constexpr bool RL = decltype(relayed)::value == 1;
                constexpr bool SAME = decltype(relayed)::value == 0;
                constexpr int TH = SAME ? THRS : THR;
                int cnt = 0;
#pragma unroll
                for (int q = 0; q < 4; ++q) {
                    const int s = RL ? (q == RQ ? 3 : q < RQ ? q : q - 1) : q;
                    u32x4 v[2];
#pragma unroll
                    for (int ks = 0; ks < 2; ++ks)
                        v[ks] = pack8(wv[ks][0][s], wv[ks][1][s], wv[ks][2][s], wv[ks][3][s], wv[ks][4][s], wv[ks][5][s], wv[ks][6][s], wv[ks][7][s], mul);
#pragma unroll
                    for (int dd = 0; dd < 4; ++dd)
#pragma unroll
                        for (int ks = 0; ks < 2; ++ks) {
                            if (RL && q == RQ && dd > 0) continue;
                            if (cnt >= TH) wait_vm<TH - 1>();
                            ++cnt;
                            __builtin_amdgcn_raw_buffer_store_b128(v[ks], (RL && q == RQ) ? rr : rw[dd], off + s * 8192u + ks * 1024u, 0, ST_AUX);
                            asm volatile("" ::: "memory");
                        }
                    if (RELAY && SAME && q == RPOS) {
#pragma unroll
                        for (int f = 0; f < 2; ++f) rl[f] = __builtin_amdgcn_raw_buffer_load_b128(rme, roff + f * 1024u, 0, 17);
                        asm volatile("" ::: "memory");
                    }
                    if (RELAY && SAME && q == RPOS + 1 && (blk >> 2) < RB) {
                        TS(10);
                        if (__builtin_expect(!__all(umin4(umin4(0xffffffffu, rl[0]), rl[1]) != 0u), 0)) {
                            Waiter W;
                            W.tmo = (unsigned*)(myws + OFF_TMO);
                            W.call = call;
                            W.t0 = __builtin_amdgcn_s_memrealtime();
                            W.ok = true;
                            do {
                                __builtin_amdgcn_s_sleep(POLL_SLEEP);
#pragma unroll
                                for (int f = 0; f < 2; ++f) rl[f] = __builtin_amdgcn_raw_buffer_load_b128(rme, roff + f * 1024u, 0, 17);
                            } while (!__all(umin4(umin4(0xffffffffu, rl[0]), rl[1]) != 0u) && !W.expired());
                        }
                        TS(9);
#pragma unroll
                        for (int dd = 0; dd < 4; ++dd) {
                            if ((((dd + wave + blk) & 3) << 1) + half == rank) continue;
#pragma unroll
                            for (int f = 0; f < 2; ++f) {
                                wait_vm<THRR - 1>();
                                __builtin_amdgcn_raw_buffer_store_b128(rl[f], rw[dd], roff + f * 1024u, 0, ST_AUX);
                                asm volatile("" ::: "memory");
                            }
                        }
                    }
                    if (q == ((RL && RQ == 0) ? 1 : 0)) {
                        unsigned m = 0;
#pragma unroll
                        for (int z = 0; z < 8; ++z) { m = umax4(m, xb[z][0]); m = umax4(m, xb[z][1]); }
                        eb = __builtin_amdgcn_readfirstlane(clamp_exp(wave_umax(m)));
                        if (lane == 0) word_store(dx + FX_OFF + (rank * 32 + 16 + rb16) * 8, call, eb);
                    }
                    TS(3 + q);
                }
            };
            if (RELAY && half != (rank & 1) && (blk >> 2) < RB) push(std::true_type{});
            else if (half != (rank & 1)) push(std::integral_constant<int, 2>{});
            else push(std::false_type{});
        }
        {
            const float mul = __builtin_bit_cast(float, (254u - eb) << 23);
            const unsigned off = XF_OFF + (unsigned)((rank * 32 + 16 + rb16) * 8) * 1024u + lane * 16u;
#pragma unroll
            for (int s = 0; s < 8; ++s) {
                const u32x4 v = pack8(xb[s][0][0], xb[s][0][1], xb[s][0][2], xb[s][0][3], xb[s][1][0], xb[s][1][1], xb[s][1][2], xb[s][1][3], mul);
                wait_vm<THRB - 1>();
                __builtin_amdgcn_raw_buffer_store_b128(v, rx, off + s * 1024u, 0, ST_AUX);
                asm volatile("" ::: "memory");
            }
        }
        TS(7);
        if (threadIdx.x == 0) __hip_atomic_store(ctr, call, __ATOMIC_RELAXED, __HIP_MEMORY_SCOPE_AGENT);
        TS_FLUSH((half != (rank & 1) ? 12 : 8) + wave);
        return;
    }

    if (g >= NPROD + NFOR + NOWN) {
        const unsigned call = ctr_wait(c0);
        const __amdgpu_buffer_rsrc_t rz = __builtin_amdgcn_make_buffer_rsrc(myws + OFF_DATA + ((call + 2u) % 3u) * BUF_BYTES, 0, (int)ZERO_BYTES, 0x00020000);
        const unsigned zo = (unsigned)((g - NPROD - NFOR - NOWN) * 8 + wave) * 57344u + lane * 16u;
#pragma unroll 1
        for (int k = 0; k < 56 * ZREP; ++k) {
            wait_vm<ZTHR - 1>();
            __builtin_amdgcn_raw_buffer_store_b128((u32x4){0u, 0u, 0u, 0u}, rz, zo + (k % 56) * 1024u, 0, ZERO_AUX);
        }
        if (threadIdx.x == 0) __hip_atomic_store(ctr, call, __ATOMIC_RELAXED, __HIP_MEMORY_SCOPE_AGENT);
        return;
    }

    if (g >= NPROD + NFOR && wave == 0) {
        const int xt = g & 7, xrb = (g - NPROD - NFOR) >> 3;
        const int rowa = MS * (2 * (xrb >> 2) + 1 - (xt & 1)) + 64 * (xt >> 1) + 16 * (xrb & 3) + lc;
        f32x4 xa[8][2];
#pragma unroll
        for (int s = 0; s < 8; ++s) {
            const float* src = px + (size_t)rowa * KS + s * 32 + lg * 8;
            xa[s][0] = *(const f32x4*)src;
            xa[s][1] = *(const f32x4*)(src + 4);
        }
        const unsigned call = ctr_wait(c0);
        char* const dx = wsx + OFF_DATA + (call % 3u) * BUF_BYTES;
        const __amdgpu_buffer_rsrc_t rx = __builtin_amdgcn_make_buffer_rsrc(dx, 0, (int)BUF_BYTES, 0x00020000);
        unsigned m = 0;
#pragma unroll
        for (int s = 0; s < 8; ++s) { m = umax4(m, xa[s][0]); m = umax4(m, xa[s][1]); }
        const unsigned ea = __builtin_amdgcn_readfirstlane(clamp_exp(wave_umax(m)));
        const float mul = __builtin_bit_cast(float, (254u - ea) << 23);
        const unsigned off = XF_OFF + (unsigned)((rank * 32 + xrb) * 8) * 1024u + lane * 16u;
#pragma unroll
        for (int s = 0; s < 8; ++s) {
            const u32x4 v = pack8(xa[s][0][0], xa[s][0][1], xa[s][0][2], xa[s][0][3], xa[s][1][0], xa[s][1][1], xa[s][1][2], xa[s][1][3], mul);
            __builtin_amdgcn_raw_buffer_store_b128(v, rx, off + s * 1024u, 0, ST_AUX);
        }
        if (lane == 0) word_store(dx + FX_OFF + (rank * 32 + xrb) * 8, call, ea);
        TS(9);
    }
    const unsigned call = ctr_wait(c0);
    const unsigned boff = OFF_DATA + (call % 3u) * BUF_BYTES;
    const int j = wave;
    const int h = rank & 1;
    const __amdgpu_buffer_rsrc_t rs = __builtin_amdgcn_make_buffer_rsrc(myws + boff, 0, (int)BUF_BYTES, 0x00020000);
    const __amdgpu_buffer_rsrc_t ro = __builtin_amdgcn_make_buffer_rsrc(P.out, 0, MS * ND * 4, 0x00020000);
    if (threadIdx.x == 0) s_bad = 0;
    Waiter W;
    W.tmo = (unsigned*)(myws + OFF_TMO);
    W.call = call;
    W.t0 = __builtin_amdgcn_s_memrealtime();
    W.ok = true;
    const float qn = __builtin_nanf("");

    if (g < NPROD + NFOR) {
        const int cg = g - NPROD;
        const int blk = (cg & 7) * 2 + ((cg >> 3) & 1);
        const int rb64 = cg >> 4;
        if (threadIdx.x < 4) s_cnt[threadIdx.x] = 0;
        __syncthreads();
        u32x4 a[32];
        acquire<32>(a, rs, XF_OFF + (unsigned)((j * 32 + rb64 * 4) * 8) * 1024u + lane * 16u, lane, W);
        TS(1);
        const unsigned bbase = WF_OFF + (unsigned)((j * 16 + blk) * 32) * 1024u + lane * 16u;
        const int dst = 2 * rb64 + 1 - h;
        const __amdgpu_buffer_rsrc_t rd = __builtin_amdgcn_make_buffer_rsrc(P.ws[dst] + boff, 0, (int)BUF_BYTES, 0x00020000);
        const unsigned roff = RB_OFF + (unsigned)(((rank >> 1) * 16 + blk) * 4) * 4096u + threadIdx.x * 16u;
        int sc[4];
        unsigned pushed = 0;
        const auto reduce = [&](unsigned p) {
            const float* src = red + p * (8 * 16 * 68) + lc * 68 + (threadIdx.x >> 4) * 4;
            f32x4 sum = *(const f32x4*)src;
#pragma unroll
            for (int k = 1; k < 8; ++k) sum += *(const f32x4*)(src + k * (16 * 68));
            if (s_bad) sum = (f32x4){qn, qn, qn, qn};
            u32x4 v = __builtin_bit_cast(u32x4, sum);
#pragma unroll
            for (int k = 0; k < 4; ++k) v[k] = v[k] ? v[k] : 0x80000000u;
            return v;
        };
        const auto complete = [&](unsigned p) {
            const unsigned c = __builtin_amdgcn_readfirstlane(__hip_atomic_load(&s_cnt[p], __ATOMIC_RELAXED, __HIP_MEMORY_SCOPE_WORKGROUP));
            asm volatile("" ::: "memory");
            return c == 8u;
        };
#pragma unroll
        for (int s = 0; s < 4; ++s) {
            u32x4 b[8];
            wait_landed<8>(rs, bbase + s * 8192u, lane, W);
#pragma unroll
            for (int f = 0; f < 8; ++f) b[f] = __builtin_amdgcn_raw_buffer_load_b128(rs, bbase + s * 8192u + f * 1024u, 0, 0);
            if (s > 0) {
                asm volatile("" ::: "memory");
                u32x4 v = (u32x4){0u, 0u, 0u, 0u};
                unsigned off = 0x80000000u;
                if (wave < 4 && complete(pushed)) {
                    v = reduce(pushed);
                    off = roff + pushed * 4096u;
                    ++pushed;
                }
                __builtin_amdgcn_raw_buffer_store_b128(v, rd, off, 0, ST_AUX);
                asm volatile("" ::: "memory");
            }
            validate<8>(b, rs, bbase + s * 8192u, Lin{}, W);
            f32x4 acc[4];
#pragma unroll
            for (int rb = 0; rb < 4; ++rb) acc[rb] = (f32x4){0.f, 0.f, 0.f, 0.f};
#pragma unroll
            for (int k = 0; k < 8; ++k)
#pragma unroll
                for (int rb = 0; rb < 4; ++rb) acc[rb] = mfma(a[rb * 8 + k], b[k], acc[rb]);
            if (s == 0) {
                const unsigned wo = lane < 4 ? FX_OFF + (unsigned)(j * 32 + rb64 * 4 + lane) * 8u : FW_OFF + (unsigned)(j * 16 + blk) * 8u;
                const unsigned v = wait_words(myws + boff + wo, call, W);
#pragma unroll
                for (int rb = 0; rb < 4; ++rb) sc[rb] = (int)__builtin_amdgcn_readlane(v, rb) + (int)__builtin_amdgcn_readlane(v, 4) - 254;
            }
            float* const slab = red + (s * 8 + j) * (16 * 68);
#pragma unroll
            for (int rb = 0; rb < 4; ++rb) {
                f32x4 v;
#pragma unroll
                for (int r = 0; r < 4; ++r) v[r] = __builtin_ldexpf(acc[rb][r], sc[rb]);
                *(f32x4*)(slab + lc * 68 + rb * 16 + lg * 4) = v;
            }
            if (!W.ok && lane == 0) s_bad = 1;
            asm volatile("" ::: "memory");
            if (lane == 0) __hip_atomic_fetch_add(&s_cnt[s], 1u, __ATOMIC_RELAXED, __HIP_MEMORY_SCOPE_WORKGROUP);
            asm volatile("" ::: "memory");
            TS(2 + s);
        }
        if (wave < 4) {
            while (pushed < 4u) {
                while (!complete(pushed)) {
                    if (W.expired()) break;
                }
                u32x4 v = reduce(pushed);
                if (!W.ok) v = __builtin_bit_cast(u32x4, (f32x4){qn, qn, qn, qn});
                __builtin_amdgcn_raw_buffer_store_b128(v, rd, roff + pushed * 4096u, 0, ST_AUX);
                ++pushed;
            }
            TS(6);
        } else {
            const unsigned tid = threadIdx.x - 256u;
            const unsigned off = RB_OFF + (unsigned)((rb64 * 16 + blk) * 4) * 4096u + tid * 16u;
            u32x4 v[4];
            for (;;) {
#pragma unroll
                for (int s = 0; s < 4; ++s) v[s] = __builtin_amdgcn_raw_buffer_load_b128(rs, off + s * 4096u, 0, 17);
                if (__all(umin4(umin4(umin4(umin4(0xffffffffu, v[0]), v[1]), v[2]), v[3]) != 0u)) break;
                if (W.expired()) break;
                __builtin_amdgcn_s_sleep(HOT_SLEEP);
            }
            const unsigned o = (unsigned)((rb64 * 64 + (tid >> 4) * 4) * ND + (1 - h) * 1024 + blk * 64 + lc * 4) * 4u;
#pragma unroll
            for (int r = 0; r < 4; ++r) {
                f32x4 q = __builtin_bit_cast(f32x4, (u32x4){v[0][r], v[1][r], v[2][r], v[3][r]});
                if (!W.ok) q = (f32x4){qn, qn, qn, qn};
                __builtin_amdgcn_raw_buffer_store_b128(__builtin_bit_cast(u32x4, q), ro, o + (unsigned)r * (ND * 4u), 0, OUT_AUX);
            }
            TS(7);
        }
        if (threadIdx.x == 0) __hip_atomic_store(ctr, call, __ATOMIC_RELAXED, __HIP_MEMORY_SCOPE_AGENT);
        TS_FLUSH(wave);
        return;
    }

    {
        const int cg = g - NPROD - NFOR;
        const int blk = (cg & 7) * 2 + ((cg >> 3) & 1);
        const int rb32 = cg >> 4;
        u32x4 b[32];
        const unsigned bbase = WF_OFF + (unsigned)((j * 16 + blk) * 32) * 1024u + lane * 16u;
#pragma unroll
        for (int s = 0; s < 4; ++s) {
            acquire<8>(b + s * 8, rs, bbase + s * 8192u, lane, W);
            TS(1 + s);
        }
        int sc[2];
        {
            const unsigned wo = lane < 2 ? FX_OFF + (unsigned)(j * 32 + 16 + rb32 * 2 + lane) * 8u : FW_OFF + (unsigned)(j * 16 + blk) * 8u;
            const unsigned v = wait_words(myws + boff + wo, call, W);
#pragma unroll
            for (int rb = 0; rb < 2; ++rb) sc[rb] = (int)__builtin_amdgcn_readlane(v, rb) + (int)__builtin_amdgcn_readlane(v, 2) - 254;
        }
        const unsigned abase = XF_OFF + (unsigned)((j * 32 + 16 + rb32 * 2) * 8) * 1024u + lane * 16u;
        f32x4 acc[2][4];
#pragma unroll
        for (int rb = 0; rb < 2; ++rb)
#pragma unroll
            for (int s = 0; s < 4; ++s) acc[rb][s] = (f32x4){0.f, 0.f, 0.f, 0.f};
        for (;;) {
            unsigned v = 1u;
            if (lane == 63) v = __builtin_amdgcn_raw_buffer_load_b32(rs, abase + 8192u + 12u, 0, 17);
            if (__all(v != 0u)) break;
            if (W.expired()) break;
            __builtin_amdgcn_s_sleep(POLL_SLEEP);
        }
        TS(5);
        u32x4 a[2][8];
        const auto aoff = [](int f) { return (unsigned)((f >> 2) * 8192 + (f & 3) * 1024); };
#pragma unroll
        for (int c = 0; c < 2; ++c)
#pragma unroll
            for (int f = 0; f < 8; ++f) a[c][f] = __builtin_amdgcn_raw_buffer_load_b128(rs, abase + c * 4096u + aoff(f), 0, 17);
#pragma unroll
        for (int c = 0; c < 2; ++c) {
            validate<8>(a[c], rs, abase + c * 4096u, aoff, W);
#pragma unroll
            for (int ks = 0; ks < 4; ++ks)
#pragma unroll
                for (int rb = 0; rb < 2; ++rb)
#pragma unroll
                    for (int s = 0; s < 4; ++s) acc[rb][s] = mfma(a[c][rb * 4 + ks], b[s * 8 + c * 4 + ks], acc[rb][s]);
        }
        TS(6);
#pragma unroll
        for (int rb = 0; rb < 2; ++rb)
#pragma unroll
            for (int r = 0; r < 4; ++r) {
                f32x4 v;
#pragma unroll
                for (int s = 0; s < 4; ++s) v[s] = __builtin_ldexpf(acc[rb][s][r], sc[rb]);
                *(f32x4*)(red + (j * 32 + rb * 16 + lg * 4 + r) * 64 + lc * 4) = v;
            }
        if (!W.ok && lane == 0) s_bad = 1;
        __syncthreads();
        TS(7);
        const float* src = red + threadIdx.x * 4;
        f32x4 sum = *(const f32x4*)src;
#pragma unroll
        for (int k = 1; k < 8; ++k) sum += *(const f32x4*)(src + k * 2048);
        if (s_bad) sum = (f32x4){qn, qn, qn, qn};
        __builtin_amdgcn_raw_buffer_store_b128(__builtin_bit_cast(u32x4, sum), ro, (unsigned)((rb32 * 32 + (threadIdx.x >> 4)) * ND + h * 1024 + blk * 64 + lc * 4) * 4u, 0, OUT_AUX);
        if (threadIdx.x == 0) __hip_atomic_store(ctr, call, __ATOMIC_RELAXED, __HIP_MEMORY_SCOPE_AGENT);
        TS(8);
        TS_FLUSH(wave);
    }
}

}

extern "C" void kernel_launch(int rank, int nranks, ncclComm_t comm,
                              hipStream_t stream,
                              void* const* d_in, const int* in_sizes, int n_in,
                              void* d_out, int out_size,
                              const int* in_dtypes, int out_dtype,
                              void* const* d_out_all,
                              void* const* d_ws_all, size_t ws_bytes) {
    if (nranks != NR || (unsigned)rank >= (unsigned)NR || ws_bytes < (size_t)WS_NEED || n_in < 2 ||
        in_sizes[0] != 2048 * KS || in_sizes[1] != KS * ND || out_size != MS * ND) {
        fprintf(stderr, "kernel_launch: unexpected problem geometry\n");
        return;
    }
    Params P;
    P.x = (const float*)d_in[0];
    P.w = (const float*)d_in[1];
    P.out = (float*)d_out;
    for (int p = 0; p < NR; ++p) P.ws[p] = (char*)d_ws_all[p];
    P.rank = rank;
    P.myws = (char*)d_ws_all[rank];
    hipLaunchKernelGGL(gemm_ag_kernel, dim3(NWG), dim3(NTHREADS), 0, stream, P);
    HIP_CHECK(hipGetLastError());
}
```

```cpp
#include <hip/hip_runtime.h>
#include <rccl/rccl.h>
#include <cstdio>
#include <cstdint>
#include <utility>
#include <type_traits>

#define HIP_CHECK(expr) do { hipError_t _e = (expr); if (_e != hipSuccess) { \
    fprintf(stderr, "HIP error %s at %s:%d\n", hipGetErrorString(_e), __FILE__, __LINE__); \
    return; } } while (0)

#define ST_AUX 17
#define THR 4
#define THRB 8
#define RELAY 1
#define THRS 5
#define THRR THRS
#define ZREP 1
#define TXS 20
#define TXO 40
#define PMAP 0
#define RB 4
#define RQ 0
#define RPOS 1
#define XL0 0
#define TS0 280
#define DS 126
#define TO0 290
#define DO 124
#define DXS 24
#define DXO 45
#define LW 3
#define LM 30
#define PFREE 5
#define RTHR 2
#define ZERO_AUX 17
#define ZTHR 2
#define POLL_SLEEP 1
#define HOT_SLEEP 1

#define TS_DECL
#define TS(slot) do { } while (0)
#define TS_FLUSH(w) do { } while (0)

namespace {

typedef _Float16 half8 __attribute__((ext_vector_type(8)));
typedef float f32x2 __attribute__((ext_vector_type(2)));
typedef float f32x4 __attribute__((ext_vector_type(4)));
typedef unsigned u32x4 __attribute__((ext_vector_type(4)));
typedef unsigned long long u64;

constexpr int NR = 8;
constexpr int ND = 2048;
constexpr int MS = 256;
constexpr int KS = 256;
constexpr int NPROD = 32;
constexpr int NFOR = 64;
constexpr int NOWN = 128;
constexpr int NJAN = 16;
constexpr int NWG = NPROD + NFOR + NOWN + NJAN;
constexpr int NTHREADS = 512;

constexpr unsigned OFF_CTR = 0;
constexpr unsigned OFF_TMO = 65536;
constexpr unsigned OFF_DATA = 1u << 20;
constexpr unsigned XF_OFF = 0;
constexpr unsigned WF_OFF = 2u << 20;
constexpr unsigned RB_OFF = 6u << 20;
constexpr unsigned ZERO_BYTES = 7u << 20;
constexpr unsigned FX_OFF = 7u << 20;
constexpr unsigned FW_OFF = FX_OFF + 4096;
constexpr unsigned BUF_BYTES = (7u << 20) + 65536;
constexpr unsigned WS_NEED = OFF_DATA + 3 * BUF_BYTES;
constexpr u64 DEADLINE_TICKS = 50000000ull;

struct Params {
    const float* x;
    const float* w;
    float* out;
    char* myws;
    char* ws[NR];
    int rank;
};

template <int N> __device__ __forceinline__ void wait_vm() { asm volatile("s_waitcnt vmcnt(%0)" ::"n"(N) : "memory"); }

__device__ __forceinline__ void wait_vm_c(int n) {
    switch (n) {
        case 0: wait_vm<0>(); break;   case 1: wait_vm<1>(); break;   case 2: wait_vm<2>(); break;   case 3: wait_vm<3>(); break;
        case 4: wait_vm<4>(); break;   case 5: wait_vm<5>(); break;   case 6: wait_vm<6>(); break;   case 7: wait_vm<7>(); break;
        case 8: wait_vm<8>(); break;   case 9: wait_vm<9>(); break;   case 10: wait_vm<10>(); break; case 11: wait_vm<11>(); break;
        case 12: wait_vm<12>(); break; case 13: wait_vm<13>(); break; case 14: wait_vm<14>(); break; default: wait_vm<15>(); break;
    }
}

template <int CTRL> __device__ __forceinline__ unsigned dpp_umax(unsigned v) {
    const unsigned t = (unsigned)__builtin_amdgcn_update_dpp(0, (int)v, CTRL, 0xf, 0xf, false);
    return v > t ? v : t;
}
__device__ __forceinline__ unsigned wave_umax(unsigned v) {
    v = dpp_umax<0xB1>(v);
    v = dpp_umax<0x4E>(v);
    v = dpp_umax<0x141>(v);
    v = dpp_umax<0x140>(v);
    unsigned a = __builtin_amdgcn_readlane(v, 0), b = __builtin_amdgcn_readlane(v, 16);
    unsigned c = __builtin_amdgcn_readlane(v, 32), d = __builtin_amdgcn_readlane(v, 48);
    a = a > b ? a : b; c = c > d ? c : d;
    return a > c ? a : c;
}
__device__ __forceinline__ unsigned absbits(float v) { return __builtin_bit_cast(unsigned, v) & 0x7fffffffu; }
__device__ __forceinline__ unsigned umax4(unsigned m, f32x4 v) {
    unsigned a = absbits(v[0]), b = absbits(v[1]), c = absbits(v[2]), d = absbits(v[3]);
    a = a > b ? a : b; c = c > d ? c : d; a = a > c ? a : c;
    return m > a ? m : a;
}
__device__ __forceinline__ unsigned clamp_exp(unsigned m) {
    unsigned e = m >> 23;
    e = e < 1u ? 1u : e;
    return e > 253u ? 253u : e;
}
__device__ __forceinline__ void word_store(char* p, unsigned call, unsigned e) {
    __hip_atomic_store((u64*)p, ((u64)call << 32) | (u64)e, __ATOMIC_RELAXED, __HIP_MEMORY_SCOPE_SYSTEM);
}
__device__ __forceinline__ u32x4 pack8(float a0, float a1, float a2, float a3, float a4, float a5, float a6, float a7, float m) {
    half8 h;
    h[0] = (_Float16)(a0 * m); h[1] = (_Float16)(a1 * m); h[2] = (_Float16)(a2 * m); h[3] = (_Float16)(a3 * m);
    h[4] = (_Float16)(a4 * m); h[5] = (_Float16)(a5 * m); h[6] = (_Float16)(a6 * m); h[7] = (_Float16)(a7 * m);
    u32x4 v = __builtin_bit_cast(u32x4, h);
#pragma unroll
    for (int k = 0; k < 4; ++k) v[k] = v[k] ? v[k] : 0x8000u;
    return v;
}
__device__ __forceinline__ unsigned umin4(unsigned m, u32x4 v) {
    unsigned a = v[0] < v[1] ? v[0] : v[1], b = v[2] < v[3] ? v[2] : v[3];
    a = a < b ? a : b;
    return m < a ? m : a;
}
__device__ __forceinline__ unsigned ctr_issue(const unsigned* p) {
    unsigned v;
    asm volatile("s_load_dword %0, %1, 0x0" : "=s"(v) : "s"(p) : "memory");
    return v;
}
__device__ __forceinline__ unsigned ctr_wait(unsigned v) {
    unsigned r;
    asm volatile("s_waitcnt lgkmcnt(0)\n\ts_add_u32 %0, %1, 1" : "=s"(r) : "s"(v) : "memory", "scc");
    return r;
}

struct Waiter {
    unsigned* tmo;
    unsigned call;
    u64 t0;
    bool ok;
    __device__ __forceinline__ bool expired() {
        if (!ok) return true;
        if (__builtin_amdgcn_s_memrealtime() - t0 > DEADLINE_TICKS) {
            __hip_atomic_store(tmo, call, __ATOMIC_RELAXED, __HIP_MEMORY_SCOPE_AGENT);
            ok = false;
            return true;
        }
        return false;
    }
};

template <int N>
__device__ __forceinline__ void wait_landed(__amdgpu_buffer_rsrc_t rs, unsigned base, int lane, Waiter& W) {
    const unsigned probe = base - lane * 16u + (unsigned)(lane < N ? lane : N - 1) * 1024u + 1020u;
    for (;;) {
        unsigned v = 1u;
        if (lane == N - 1) v = __builtin_amdgcn_raw_buffer_load_b32(rs, probe, 0, 17);
        if (__all(v != 0u)) break;
        if (W.expired()) return;
        __builtin_amdgcn_s_sleep(POLL_SLEEP);
    }
}
template <int N, class OFF>
__device__ __forceinline__ void validate(u32x4* r, __amdgpu_buffer_rsrc_t rs, unsigned base, OFF off, Waiter& W) {
    unsigned m = 0xffffffffu;
#pragma unroll
    for (int f = 0; f < N; ++f) m = umin4(m, r[f]);
    while (__builtin_expect(!__all(m != 0u), 0)) {
        if (W.expired()) break;
#pragma unroll
        for (int f = 0; f < N; ++f) r[f] = __builtin_amdgcn_raw_buffer_load_b128(rs, base + off(f), 0, 17);
        m = 0xffffffffu;
#pragma unroll
        for (int f = 0; f < N; ++f) m = umin4(m, r[f]);
    }
}
struct Lin { __device__ __forceinline__ unsigned operator()(int f) const { return (unsigned)f * 1024u; } };

template <int N>
__device__ __forceinline__ void acquire(u32x4* r, __amdgpu_buffer_rsrc_t rs, unsigned base, int lane, Waiter& W) {
    wait_landed<N>(rs, base, lane, W);
#pragma unroll
    for (int f = 0; f < N; ++f) r[f] = __builtin_amdgcn_raw_buffer_load_b128(rs, base + f * 1024u, 0, 0);
    validate<N>(r, rs, base, Lin{}, W);
}

#define OUT_AUX 16
__device__ __forceinline__ f32x4 mfma(u32x4 a, u32x4 b, f32x4 c) {
    return __builtin_amdgcn_mfma_f32_16x16x32_f16(__builtin_bit_cast(half8, a), __builtin_bit_cast(half8, b), c, 0, 0, 0);
}

__device__ __forceinline__ unsigned wait_words(const char* p, unsigned call, Waiter& W) {
    u64 v;
    for (;;) {
        v = __hip_atomic_load((const u64*)p, __ATOMIC_RELAXED, __HIP_MEMORY_SCOPE_SYSTEM);
        if (__all((unsigned)(v >> 32) == call)) break;
        if (W.expired()) break;
        __builtin_amdgcn_s_sleep(2);
    }
    return (unsigned)v;
}

__global__ void __launch_bounds__(NTHREADS) gemm_ag_kernel(Params P) {
    __shared__ __attribute__((aligned(16))) float red[4 * 8 * 16 * 68];
    __shared__ unsigned s_m[4];
    __shared__ unsigned s_cnt[4];
    __shared__ int s_bad;

    const int g = blockIdx.x;
    const int wave = __builtin_amdgcn_readfirstlane(threadIdx.x >> 6);
    const int lane = threadIdx.x & 63;
    const int lg = lane >> 4, lc = lane & 15;
    const int rank = P.rank;
    char* const myws = P.myws;
    unsigned* ctr = (unsigned*)(myws + OFF_CTR + g * 128);
    TS_DECL;
    TS(0);
    const float* const px = P.x;
    const float* const pw = P.w;
    char* const wsx = P.ws[g < NPROD ? ((g >> 1) & 3) * 2 + (g & 1) : (g & 7)];
    asm volatile("" :: "s"(px), "s"(pw), "s"(myws), "s"(wsx), "s"(rank));
    const unsigned c0 = ctr_issue(ctr);
    const u64 tstart = __builtin_amdgcn_s_memrealtime();

    if (g < NPROD) {
        if (wave >= 4) return;
        const int half = g & 1, blk = g >> 1;
        const int t = (blk & 3) * 2 + half;
        const int rb16 = (blk >> 2) * 4 + wave;
        const int rowb = MS * t + 16 * rb16 + lc;
        f32x4 xb[8][2], wv[2][8];
        {
            const float* wsrc = pw + (size_t)(wave * 64 + lg * 8) * ND + half * 1024 + blk * 64 + lc * 4;
#pragma unroll
            for (int ks = 0; ks < 2; ++ks)
#pragma unroll
                for (int jj = 0; jj < 8; ++jj) wv[ks][jj] = *(const f32x4*)(wsrc + (size_t)(ks * 32 + jj) * ND);
        }
        asm volatile("" ::: "memory");
        const float* const xsrc = px + (size_t)rowb * KS + lg * 8;
        const unsigned call = ctr_wait(c0);
        const unsigned boff = OFF_DATA + (call % 3u) * BUF_BYTES;
        char* const dx = wsx + boff;
        const __amdgpu_buffer_rsrc_t rx = __builtin_amdgcn_make_buffer_rsrc(dx, 0, (int)BUF_BYTES, 0x00020000);
        {
            unsigned m = 0;
#pragma unroll
            for (int ks = 0; ks < 2; ++ks)
#pragma unroll
                for (int jj = 0; jj < 8; ++jj) m = umax4(m, wv[ks][jj]);
            m = wave_umax(m);
            if (lane == 0) s_m[wave] = m;
        }
        TS(1);
        asm volatile("s_waitcnt lgkmcnt(0)\n\ts_barrier" ::: "memory");
        unsigned ew;
        {
            unsigned m = s_m[0];
#pragma unroll
            for (int k = 1; k < 4; ++k) m = s_m[k] > m ? s_m[k] : m;
            ew = __builtin_amdgcn_readfirstlane(clamp_exp(m));
        }
        if (lane == 0) word_store(P.ws[wave * 2 + half] + boff + FW_OFF + (rank * 16 + blk) * 8, call, ew);
        __amdgpu_buffer_rsrc_t rw[4];
#pragma unroll
        for (int dd = 0; dd < 4; ++dd) {
            const int d = (((dd + wave + blk) & 3) << 1) + half;
            rw[dd] = __builtin_amdgcn_make_buffer_rsrc(P.ws[d] + boff, 0, (int)BUF_BYTES, 0x00020000);
        }
        unsigned eb = 0;
        int xi = 0;
        const unsigned t32 = (unsigned)tstart;
        TS(2);
        {
            const float mul = __builtin_bit_cast(float, (254u - ew) << 23);
            const unsigned off = WF_OFF + (unsigned)((rank * 16 + blk) * 32 + wave * 2) * 1024u + lane * 16u;
            const __amdgpu_buffer_rsrc_t rr = __builtin_amdgcn_make_buffer_rsrc(P.ws[((blk & 3) << 1) + half] + boff, 0, (int)BUF_BYTES, 0x00020000);
            const __amdgpu_buffer_rsrc_t rme = __builtin_amdgcn_make_buffer_rsrc(myws + boff, 0, (int)BUF_BYTES, 0x00020000);
            const unsigned roff = WF_OFF + (unsigned)(((2 * (blk & 3) + 1 - half) * 16 + (blk & 12) + (rank >> 1)) * 32 + 24 + wave * 2) * 1024u + lane * 16u;
            u32x4 rl[2];
            const auto pace = [&](bool same, int i, int n) {
                wait_vm_c(n + LW);
                const unsigned due = (unsigned)((same ? TS0 : TO0) + i * (same ? DS : DO) / 4);
                const unsigned now = (unsigned)__builtin_amdgcn_s_memrealtime() - t32;
                if (now <= due + LM) {
                    wait_vm_c(n);
                    if (i >= PFREE)
                        while ((unsigned)__builtin_amdgcn_s_memrealtime() - t32 < due) __builtin_amdgcn_s_sleep(1);
                }
            };
            const auto push = [&](auto relayed) {
                constexpr bool RL = decltype(relayed)::value == 1;
                constexpr bool SAME = decltype(relayed)::value == 0;
                constexpr int TH = SAME ? THRS : THR;
                int cnt = 0, extra = 0;
#pragma unroll
                for (int q = 0; q < 4; ++q) {
                    const int s = RL ? (q == RQ ? 3 : q < RQ ? q : q - 1) : q;
                    u32x4 v[2];
#pragma unroll
                    for (int ks = 0; ks < 2; ++ks)
                        v[ks] = pack8(wv[ks][0][s], wv[ks][1][s], wv[ks][2][s], wv[ks][3][s], wv[ks][4][s], wv[ks][5][s], wv[ks][6][s], wv[ks][7][s], mul);
#pragma unroll
                    for (int dd = 0; dd < 4; ++dd)
#pragma unroll
                        for (int ks = 0; ks < 2; ++ks) {
                            if (RL && q == RQ && dd > 0) continue;
                            if (cnt >= TH) {
                                const int lo = cnt - TH > XL0 ? cnt - TH : XL0, hi = cnt < XL0 + 16 ? cnt : XL0 + 16;
                                pace(SAME, cnt + extra, TH - 1 + (hi > lo ? hi - lo : 0));
                            }
                            __builtin_amdgcn_raw_buffer_store_b128(v[ks], (RL && q == RQ) ? rr : rw[dd], off + s * 8192u + ks * 1024u, 0, ST_AUX);
                            asm volatile("" ::: "memory");
                            if (cnt >= XL0 && cnt < XL0 + 16) {
                                const int z = cnt - XL0;
                                xb[z >> 1][z & 1] = *(const f32x4*)(xsrc + (z >> 1) * 32 + (z & 1) * 4);
                                asm volatile("" ::: "memory");
                            }
                            ++cnt;
                        }
                    if (RELAY && SAME && q == RPOS) {
#pragma unroll
                        for (int f = 0; f < 2; ++f) rl[f] = __builtin_amdgcn_raw_buffer_load_b128(rme, roff + f * 1024u, 0, 17);
                        asm volatile("" ::: "memory");
                    }
                    if (RELAY && SAME && q == RPOS + 1 && (blk >> 2) < RB) {
                        TS(10);
                        asm volatile("" : "+v"(rl[0]), "+v"(rl[1]));
                        if (__builtin_expect(!__all(umin4(umin4(0xffffffffu, rl[0]), rl[1]) != 0u), 0)) {
                            Waiter W;
                            W.tmo = (unsigned*)(myws + OFF_TMO);
                            W.call = call;
                            W.t0 = __builtin_amdgcn_s_memrealtime();
                            W.ok = true;
                            do {
                                __builtin_amdgcn_s_sleep(POLL_SLEEP);
#pragma unroll
                                for (int f = 0; f < 2; ++f) rl[f] = __builtin_amdgcn_raw_buffer_load_b128(rme, roff + f * 1024u, 0, 17);
                            } while (!__all(umin4(umin4(0xffffffffu, rl[0]), rl[1]) != 0u) && !W.expired());
                        }
                        TS(9);
#pragma unroll
                        for (int dd = 0; dd < 4; ++dd) {
                            if ((((dd + wave + blk) & 3) << 1) + half == rank) continue;
#pragma unroll
                            for (int f = 0; f < 2; ++f) {
                                pace(true, cnt + (dd * 3 >> 1) + f, THRR - 1);
                                __builtin_amdgcn_raw_buffer_store_b128(rl[f], rw[dd], roff + f * 1024u, 0, ST_AUX);
                                asm volatile("" ::: "memory");
                            }
                        }
                    }
                    if (RELAY && SAME && q == RPOS + 1) extra = RB * 6 / 4;
                    xi = cnt + extra;
                    if (q == (RL ? 3 : 2)) {
                        unsigned m = 0;
#pragma unroll
                        for (int z = 0; z < 8; ++z) { m = umax4(m, xb[z][0]); m = umax4(m, xb[z][1]); }
                        eb = __builtin_amdgcn_readfirstlane(clamp_exp(wave_umax(m)));
                        if (lane == 0) word_store(dx + FX_OFF + (rank * 32 + 16 + rb16) * 8, call, eb);
                    }
                    TS(3 + q);
                }
            };
            if (RELAY && half != (rank & 1) && (blk >> 2) < RB) push(std::true_type{});
            else if (half != (rank & 1)) push(std::integral_constant<int, 2>{});
            else push(std::false_type{});
        }
        unsigned xdue = half != (rank & 1) ? (unsigned)(TO0 + xi * DO / 4 + TXO) : (unsigned)(TS0 + xi * DS / 4 + TXS);
        const unsigned xstep = half != (rank & 1) ? DXO : DXS;
        {
            const float mul = __builtin_bit_cast(float, (254u - eb) << 23);
            const unsigned off = XF_OFF + (unsigned)((rank * 32 + 16 + rb16) * 8) * 1024u + lane * 16u;
#pragma unroll
            for (int s = 0; s < 8; ++s) {
                const u32x4 v = pack8(xb[s][0][0], xb[s][0][1], xb[s][0][2], xb[s][0][3], xb[s][1][0], xb[s][1][1], xb[s][1][2], xb[s][1][3], mul);
                wait_vm<THRB - 1>();
                while ((unsigned)__builtin_amdgcn_s_memrealtime() - t32 < xdue) __builtin_amdgcn_s_sleep(1);
                xdue += xstep;
                __builtin_amdgcn_raw_buffer_store_b128(v, rx, off + s * 1024u, 0, ST_AUX);
                asm volatile("" ::: "memory");
            }
        }
        TS(7);
        if (threadIdx.x == 0) __hip_atomic_store(ctr, call, __ATOMIC_RELAXED, __HIP_MEMORY_SCOPE_AGENT);
        TS_FLUSH((half != (rank & 1) ? 12 : 8) + wave);
        return;
    }

    if (g >= NPROD + NFOR + NOWN) {
        const unsigned call = ctr_wait(c0);
        const __amdgpu_buffer_rsrc_t rz = __builtin_amdgcn_make_buffer_rsrc(myws + OFF_DATA + ((call + 2u) % 3u) * BUF_BYTES, 0, (int)ZERO_BYTES, 0x00020000);
        const unsigned zo = (unsigned)((g - NPROD - NFOR - NOWN) * 8 + wave) * 57344u + lane * 16u;
#pragma unroll 1
        for (int k = 0; k < 56 * ZREP; ++k) {
            wait_vm<ZTHR - 1>();
            __builtin_amdgcn_raw_buffer_store_b128((u32x4){0u, 0u, 0u, 0u}, rz, zo + (k % 56) * 1024u, 0, ZERO_AUX);
        }
        if (threadIdx.x == 0) __hip_atomic_store(ctr, call, __ATOMIC_RELAXED, __HIP_MEMORY_SCOPE_AGENT);
        return;
    }

    if (g >= NPROD + NFOR && wave == 0) {
        const int xt = g & 7, xrb = (g - NPROD - NFOR) >> 3;
        const int rowa = MS * (2 * (xrb >> 2) + 1 - (xt & 1)) + 64 * (xt >> 1) + 16 * (xrb & 3) + lc;
        f32x4 xa[8][2];
#pragma unroll
        for (int s = 0; s < 8; ++s) {
            const float* src = px + (size_t)rowa * KS + s * 32 + lg * 8;
            xa[s][0] = *(const f32x4*)src;
            xa[s][1] = *(const f32x4*)(src + 4);
        }
        const unsigned call = ctr_wait(c0);
        char* const dx = wsx + OFF_DATA + (call % 3u) * BUF_BYTES;
        const __amdgpu_buffer_rsrc_t rx = __builtin_amdgcn_make_buffer_rsrc(dx, 0, (int)BUF_BYTES, 0x00020000);
        unsigned m = 0;
#pragma unroll
        for (int s = 0; s < 8; ++s) { m = umax4(m, xa[s][0]); m = umax4(m, xa[s][1]); }
        const unsigned ea = __builtin_amdgcn_readfirstlane(clamp_exp(wave_umax(m)));
        const float mul = __builtin_bit_cast(float, (254u - ea) << 23);
        const unsigned off = XF_OFF + (unsigned)((rank * 32 + xrb) * 8) * 1024u + lane * 16u;
#pragma unroll
        for (int s = 0; s < 8; ++s) {
            const u32x4 v = pack8(xa[s][0][0], xa[s][0][1], xa[s][0][2], xa[s][0][3], xa[s][1][0], xa[s][1][1], xa[s][1][2], xa[s][1][3], mul);
            __builtin_amdgcn_raw_buffer_store_b128(v, rx, off + s * 1024u, 0, ST_AUX);
        }
        if (lane == 0) word_store(dx + FX_OFF + (rank * 32 + xrb) * 8, call, ea);
        TS(9);
    }
    const unsigned call = ctr_wait(c0);
    const unsigned boff = OFF_DATA + (call % 3u) * BUF_BYTES;
    const int j = wave;
    const int h = rank & 1;
    const __amdgpu_buffer_rsrc_t rs = __builtin_amdgcn_make_buffer_rsrc(myws + boff, 0, (int)BUF_BYTES, 0x00020000);
    const __amdgpu_buffer_rsrc_t ro = __builtin_amdgcn_make_buffer_rsrc(P.out, 0, MS * ND * 4, 0x00020000);
    if (threadIdx.x == 0) s_bad = 0;
    Waiter W;
    W.tmo = (unsigned*)(myws + OFF_TMO);
    W.call = call;
    W.t0 = __builtin_amdgcn_s_memrealtime();
    W.ok = true;
    const float qn = __builtin_nanf("");

    if (g < NPROD + NFOR) {
        const int cg = g - NPROD;
        const int blk = (cg & 7) * 2 + ((cg >> 3) & 1);
        const int rb64 = cg >> 4;
        if (threadIdx.x < 4) s_cnt[threadIdx.x] = 0;
        __syncthreads();
        u32x4 a[32];
        acquire<32>(a, rs, XF_OFF + (unsigned)((j * 32 + rb64 * 4) * 8) * 1024u + lane * 16u, lane, W);
        TS(1);
        const unsigned bbase = WF_OFF + (unsigned)((j * 16 + blk) * 32) * 1024u + lane * 16u;
        const int dst = 2 * rb64 + 1 - h;
        const __amdgpu_buffer_rsrc_t rd = __builtin_amdgcn_make_buffer_rsrc(P.ws[dst] + boff, 0, (int)BUF_BYTES, 0x00020000);
        const unsigned roff = RB_OFF + (unsigned)(((rank >> 1) * 16 + blk) * 4) * 4096u + threadIdx.x * 16u;
        int sc[4];
        unsigned pushed = 0;
        const auto reduce = [&](unsigned p) {
            const float* src = red + p * (8 * 16 * 68) + lc * 68 + (threadIdx.x >> 4) * 4;
            f32x4 sum = *(const f32x4*)src;
#pragma unroll
            for (int k = 1; k < 8; ++k) sum += *(const f32x4*)(src + k * (16 * 68));
            if (s_bad) sum = (f32x4){qn, qn, qn, qn};
            u32x4 v = __builtin_bit_cast(u32x4, sum);
#pragma unroll
            for (int k = 0; k < 4; ++k) v[k] = v[k] ? v[k] : 0x80000000u;
            return v;
        };
        const auto complete = [&](unsigned p) {
            const unsigned c = __builtin_amdgcn_readfirstlane(__hip_atomic_load(&s_cnt[p], __ATOMIC_RELAXED, __HIP_MEMORY_SCOPE_WORKGROUP));
            asm volatile("" ::: "memory");
            return c == 8u;
        };
#pragma unroll
        for (int s = 0; s < 4; ++s) {
            u32x4 b[8];
            wait_landed<8>(rs, bbase + s * 8192u, lane, W);
#pragma unroll
            for (int f = 0; f < 8; ++f) b[f] = __builtin_amdgcn_raw_buffer_load_b128(rs, bbase + s * 8192u + f * 1024u, 0, 0);
            if (s > 0) {
                asm volatile("" ::: "memory");
                u32x4 v = (u32x4){0u, 0u, 0u, 0u};
                unsigned off = 0x80000000u;
                if (wave < 4 && complete(pushed)) {
                    v = reduce(pushed);
                    off = roff + pushed * 4096u;
                    ++pushed;
                }
                __builtin_amdgcn_raw_buffer_store_b128(v, rd, off, 0, ST_AUX);
                asm volatile("" ::: "memory");
            }
            validate<8>(b, rs, bbase + s * 8192u, Lin{}, W);
            f32x4 acc[4];
#pragma unroll
            for (int rb = 0; rb < 4; ++rb) acc[rb] = (f32x4){0.f, 0.f, 0.f, 0.f};
#pragma unroll
            for (int k = 0; k < 8; ++k)
#pragma unroll
                for (int rb = 0; rb < 4; ++rb) acc[rb] = mfma(a[rb * 8 + k], b[k], acc[rb]);
            if (s == 0) {
                const unsigned wo = lane < 4 ? FX_OFF + (unsigned)(j * 32 + rb64 * 4 + lane) * 8u : FW_OFF + (unsigned)(j * 16 + blk) * 8u;
                const unsigned v = wait_words(myws + boff + wo, call, W);
#pragma unroll
                for (int rb = 0; rb < 4; ++rb) sc[rb] = (int)__builtin_amdgcn_readlane(v, rb) + (int)__builtin_amdgcn_readlane(v, 4) - 254;
            }
            float* const slab = red + (s * 8 + j) * (16 * 68);
#pragma unroll
            for (int rb = 0; rb < 4; ++rb) {
                f32x4 v;
#pragma unroll
                for (int r = 0; r < 4; ++r) v[r] = __builtin_ldexpf(acc[rb][r], sc[rb]);
                *(f32x4*)(slab + lc * 68 + rb * 16 + lg * 4) = v;
            }
            if (!W.ok && lane == 0) s_bad = 1;
            asm volatile("" ::: "memory");
            if (lane == 0) __hip_atomic_fetch_add(&s_cnt[s], 1u, __ATOMIC_RELAXED, __HIP_MEMORY_SCOPE_WORKGROUP);
            asm volatile("" ::: "memory");
            TS(2 + s);
        }
        if (wave < 4) {
            while (pushed < 4u) {
                while (!complete(pushed)) {
                    if (W.expired()) break;
                }
                u32x4 v = reduce(pushed);
                if (!W.ok) v = __builtin_bit_cast(u32x4, (f32x4){qn, qn, qn, qn});
                __builtin_amdgcn_raw_buffer_store_b128(v, rd, roff + pushed * 4096u, 0, ST_AUX);
                ++pushed;
            }
            TS(6);
        } else {
            const unsigned tid = threadIdx.x - 256u;
            const unsigned off = RB_OFF + (unsigned)((rb64 * 16 + blk) * 4) * 4096u + tid * 16u;
            u32x4 v[4];
            for (;;) {
#pragma unroll
                for (int s = 0; s < 4; ++s) v[s] = __builtin_amdgcn_raw_buffer_load_b128(rs, off + s * 4096u, 0, 17);
                if (__all(umin4(umin4(umin4(umin4(0xffffffffu, v[0]), v[1]), v[2]), v[3]) != 0u)) break;
                if (W.expired()) break;
                __builtin_amdgcn_s_sleep(HOT_SLEEP);
            }
            const unsigned o = (unsigned)((rb64 * 64 + (tid >> 4) * 4) * ND + (1 - h) * 1024 + blk * 64 + lc * 4) * 4u;
#pragma unroll
            for (int r = 0; r < 4; ++r) {
                f32x4 q = __builtin_bit_cast(f32x4, (u32x4){v[0][r], v[1][r], v[2][r], v[3][r]});
                if (!W.ok) q = (f32x4){qn, qn, qn, qn};
                __builtin_amdgcn_raw_buffer_store_b128(__builtin_bit_cast(u32x4, q), ro, o + (unsigned)r * (ND * 4u), 0, OUT_AUX);
            }
            TS(7);
        }
        if (threadIdx.x == 0) __hip_atomic_store(ctr, call, __ATOMIC_RELAXED, __HIP_MEMORY_SCOPE_AGENT);
        TS_FLUSH(wave);
        return;
    }

    {
        const int cg = g - NPROD - NFOR;
        const int blk = (cg & 7) * 2 + ((cg >> 3) & 1);
        const int rb32 = cg >> 4;
        u32x4 b[32];
        const unsigned bbase = WF_OFF + (unsigned)((j * 16 + blk) * 32) * 1024u + lane * 16u;
#pragma unroll
        for (int s = 0; s < 4; ++s) {
            acquire<8>(b + s * 8, rs, bbase + s * 8192u, lane, W);
            TS(1 + s);
        }
        int sc[2];
        {
            const unsigned wo = lane < 2 ? FX_OFF + (unsigned)(j * 32 + 16 + rb32 * 2 + lane) * 8u : FW_OFF + (unsigned)(j * 16 + blk) * 8u;
            const unsigned v = wait_words(myws + boff + wo, call, W);
#pragma unroll
            for (int rb = 0; rb < 2; ++rb) sc[rb] = (int)__builtin_amdgcn_readlane(v, rb) + (int)__builtin_amdgcn_readlane(v, 2) - 254;
        }
        const unsigned abase = XF_OFF + (unsigned)((j * 32 + 16 + rb32 * 2) * 8) * 1024u + lane * 16u;
        f32x4 acc[2][4];
#pragma unroll
        for (int rb = 0; rb < 2; ++rb)
#pragma unroll
            for (int s = 0; s < 4; ++s) acc[rb][s] = (f32x4){0.f, 0.f, 0.f, 0.f};
        for (;;) {
            unsigned v = 1u;
            if (lane == 63) v = __builtin_amdgcn_raw_buffer_load_b32(rs, abase + 8192u + 12u, 0, 17);
            if (__all(v != 0u)) break;
            if (W.expired()) break;
            __builtin_amdgcn_s_sleep(POLL_SLEEP);
        }
        TS(5);
        u32x4 a[2][8];
        const auto aoff = [](int f) { return (unsigned)((f >> 2) * 8192 + (f & 3) * 1024); };
#pragma unroll
        for (int c = 0; c < 2; ++c)
#pragma unroll
            for (int f = 0; f < 8; ++f) a[c][f] = __builtin_amdgcn_raw_buffer_load_b128(rs, abase + c * 4096u + aoff(f), 0, 17);
#pragma unroll
        for (int c = 0; c < 2; ++c) {
            validate<8>(a[c], rs, abase + c * 4096u, aoff, W);
#pragma unroll
            for (int ks = 0; ks < 4; ++ks)
#pragma unroll
                for (int rb = 0; rb < 2; ++rb)
#pragma unroll
                    for (int s = 0; s < 4; ++s) acc[rb][s] = mfma(a[c][rb * 4 + ks], b[s * 8 + c * 4 + ks], acc[rb][s]);
        }
        TS(6);
#pragma unroll
        for (int rb = 0; rb < 2; ++rb)
#pragma unroll
            for (int r = 0; r < 4; ++r) {
                f32x4 v;
#pragma unroll
                for (int s = 0; s < 4; ++s) v[s] = __builtin_ldexpf(acc[rb][s][r], sc[rb]);
                *(f32x4*)(red + (j * 32 + rb * 16 + lg * 4 + r) * 64 + lc * 4) = v;
            }
        if (!W.ok && lane == 0) s_bad = 1;
        __syncthreads();
        TS(7);
        const float* src = red + threadIdx.x * 4;
        f32x4 sum = *(const f32x4*)src;
#pragma unroll
        for (int k = 1; k < 8; ++k) sum += *(const f32x4*)(src + k * 2048);
        if (s_bad) sum = (f32x4){qn, qn, qn, qn};
        __builtin_amdgcn_raw_buffer_store_b128(__builtin_bit_cast(u32x4, sum), ro, (unsigned)((rb32 * 32 + (threadIdx.x >> 4)) * ND + h * 1024 + blk * 64 + lc * 4) * 4u, 0, OUT_AUX);
        if (threadIdx.x == 0) __hip_atomic_store(ctr, call, __ATOMIC_RELAXED, __HIP_MEMORY_SCOPE_AGENT);
        TS(8);
        TS_FLUSH(wave);
    }
}

}

extern "C" void kernel_launch(int rank, int nranks, ncclComm_t comm,
                              hipStream_t stream,
                              void* const* d_in, const int* in_sizes, int n_in,
                              void* d_out, int out_size,
                              const int* in_dtypes, int out_dtype,
                              void* const* d_out_all,
                              void* const* d_ws_all, size_t ws_bytes) {
    if (nranks != NR || (unsigned)rank >= (unsigned)NR || ws_bytes < (size_t)WS_NEED || n_in < 2 ||
        in_sizes[0] != 2048 * KS || in_sizes[1] != KS * ND || out_size != MS * ND) {
        fprintf(stderr, "kernel_launch: unexpected problem geometry\n");
        return;
    }
    Params P;
    P.x = (const float*)d_in[0];
    P.w = (const float*)d_in[1];
    P.out = (float*)d_out;
    for (int p = 0; p < NR; ++p) P.ws[p] = (char*)d_ws_all[p];
    P.rank = rank;
    P.myws = (char*)d_ws_all[rank];
    hipLaunchKernelGGL(gemm_ag_kernel, dim3(NWG), dim3(NTHREADS), 0, stream, P);
    HIP_CHECK(hipGetLastError());
}
```

```cpp
#include <hip/hip_runtime.h>
#include <rccl/rccl.h>
#include <cstdio>
#include <cstdint>
#include <utility>
#include <type_traits>

#define HIP_CHECK(expr) do { hipError_t _e = (expr); if (_e != hipSuccess) { \
    fprintf(stderr, "HIP error %s at %s:%d\n", hipGetErrorString(_e), __FILE__, __LINE__); \
    return; } } while (0)

#define ST_AUX 17
#define THR 4
#define THRB 8
#define RELAY 1
#define THRS 5
#define THRR THRS
#define ZREP 1
#define TXS 20
#define TXO 40
#define PMAP 0
#define RB 4
#define RQ 0
#define RPOS 1
#define XL0 0
#define TS0 280
#define DS 126
#define TO0 290
#define DO 124
#define DXS 24
#define DXO 45
#define LW 3
#define LM 30
#define PFREE 5
#define RTHR 2
#define THF 8
#define TCHK 350
#define NLATE 3
#define HELP 2
#define SKIP 1
#define ZERO_AUX 17
#define ZTHR 4
#define POLL_SLEEP 1
#define HOT_SLEEP 1

#define TS_DECL
#define TS(slot) do { } while (0)
#define TS_FLUSH(w) do { } while (0)

namespace {

typedef _Float16 half8 __attribute__((ext_vector_type(8)));
typedef float f32x2 __attribute__((ext_vector_type(2)));
typedef float f32x4 __attribute__((ext_vector_type(4)));
typedef unsigned u32x4 __attribute__((ext_vector_type(4)));
typedef unsigned long long u64;

constexpr int NR = 8;
constexpr int ND = 2048;
constexpr int MS = 256;
constexpr int KS = 256;
constexpr int NPROD = 32;
constexpr int NFOR = 64;
constexpr int NOWN = 128;
constexpr int NJAN = 16;
constexpr int NHLP = 8;
constexpr int NWG = NPROD + NFOR + NOWN + NJAN + NHLP;
constexpr int NTHREADS = 512;

constexpr unsigned OFF_CTR = 0;
constexpr unsigned OFF_TMO = 65536;
constexpr unsigned OFF_HELLO = 65536 + 4096;
constexpr unsigned OFF_PLEDGE = 65536 + 8192;
constexpr unsigned OFF_DATA = 1u << 20;
constexpr unsigned XF_OFF = 0;
constexpr unsigned WF_OFF = 2u << 20;
constexpr unsigned RB_OFF = 6u << 20;
constexpr unsigned ZERO_BYTES = 7u << 20;
constexpr unsigned FX_OFF = 7u << 20;
constexpr unsigned FW_OFF = FX_OFF + 4096;
constexpr unsigned BUF_BYTES = (7u << 20) + 65536;
constexpr unsigned WS_NEED = OFF_DATA + 3 * BUF_BYTES;
constexpr u64 DEADLINE_TICKS = 50000000ull;

struct Params {
    const float* x;
    const float* w;
    float* out;
    char* myws;
    char* ws[NR];
    int rank;
};

template <int N> __device__ __forceinline__ void wait_vm() { asm volatile("s_waitcnt vmcnt(%0)" ::"n"(N) : "memory"); }

__device__ __forceinline__ void wait_vm_c(int n) {
    switch (n) {
        case 0: wait_vm<0>(); break;   case 1: wait_vm<1>(); break;   case 2: wait_vm<2>(); break;   case 3: wait_vm<3>(); break;
        case 4: wait_vm<4>(); break;   case 5: wait_vm<5>(); break;   case 6: wait_vm<6>(); break;   case 7: wait_vm<7>(); break;
        case 8: wait_vm<8>(); break;   case 9: wait_vm<9>(); break;   case 10: wait_vm<10>(); break; case 11: wait_vm<11>(); break;
        case 12: wait_vm<12>(); break; case 13: wait_vm<13>(); break; case 14: wait_vm<14>(); break; default: wait_vm<15>(); break;
    }
}

template <int CTRL> __device__ __forceinline__ unsigned dpp_umax(unsigned v) {
    const unsigned t = (unsigned)__builtin_amdgcn_update_dpp(0, (int)v, CTRL, 0xf, 0xf, false);
    return v > t ? v : t;
}
__device__ __forceinline__ unsigned wave_umax(unsigned v) {
    v = dpp_umax<0xB1>(v);
    v = dpp_umax<0x4E>(v);
    v = dpp_umax<0x141>(v);
    v = dpp_umax<0x140>(v);
    unsigned a = __builtin_amdgcn_readlane(v, 0), b = __builtin_amdgcn_readlane(v, 16);
    unsigned c = __builtin_amdgcn_readlane(v, 32), d = __builtin_amdgcn_readlane(v, 48);
    a = a > b ? a : b; c = c > d ? c : d;
    return a > c ? a : c;
}
__device__ __forceinline__ unsigned absbits(float v) { return __builtin_bit_cast(unsigned, v) & 0x7fffffffu; }
__device__ __forceinline__ unsigned umax4(unsigned m, f32x4 v) {
    unsigned a = absbits(v[0]), b = absbits(v[1]), c = absbits(v[2]), d = absbits(v[3]);
    a = a > b ? a : b; c = c > d ? c : d; a = a > c ? a : c;
    return m > a ? m : a;
}
__device__ __forceinline__ unsigned clamp_exp(unsigned m) {
    unsigned e = m >> 23;
    e = e < 1u ? 1u : e;
    return e > 253u ? 253u : e;
}
__device__ __forceinline__ void word_store(char* p, unsigned call, unsigned e) {
    __hip_atomic_store((u64*)p, ((u64)call << 32) | (u64)e, __ATOMIC_RELAXED, __HIP_MEMORY_SCOPE_SYSTEM);
}
__device__ __forceinline__ u32x4 pack8(float a0, float a1, float a2, float a3, float a4, float a5, float a6, float a7, float m) {
    half8 h;
    h[0] = (_Float16)(a0 * m); h[1] = (_Float16)(a1 * m); h[2] = (_Float16)(a2 * m); h[3] = (_Float16)(a3 * m);
    h[4] = (_Float16)(a4 * m); h[5] = (_Float16)(a5 * m); h[6] = (_Float16)(a6 * m); h[7] = (_Float16)(a7 * m);
    u32x4 v = __builtin_bit_cast(u32x4, h);
#pragma unroll
    for (int k = 0; k < 4; ++k) v[k] = v[k] ? v[k] : 0x8000u;
    return v;
}
__device__ __forceinline__ unsigned umin4(unsigned m, u32x4 v) {
    unsigned a = v[0] < v[1] ? v[0] : v[1], b = v[2] < v[3] ? v[2] : v[3];
    a = a < b ? a : b;
    return m < a ? m : a;
}
__device__ __forceinline__ unsigned ctr_issue(const unsigned* p) {
    unsigned v;
    asm volatile("s_load_dword %0, %1, 0x0" : "=s"(v) : "s"(p) : "memory");
    return v;
}
__device__ __forceinline__ unsigned ctr_wait(unsigned v) {
    unsigned r;
    asm volatile("s_waitcnt lgkmcnt(0)\n\ts_add_u32 %0, %1, 1" : "=s"(r) : "s"(v) : "memory", "scc");
    return r;
}

struct Waiter {
    unsigned* tmo;
    unsigned call;
    u64 t0;
    bool ok;
    __device__ __forceinline__ bool expired() {
        if (!ok) return true;
        if (__builtin_amdgcn_s_memrealtime() - t0 > DEADLINE_TICKS) {
            __hip_atomic_store(tmo, call, __ATOMIC_RELAXED, __HIP_MEMORY_SCOPE_AGENT);
            ok = false;
            return true;
        }
        return false;
    }
};

template <int N>
__device__ __forceinline__ void wait_landed(__amdgpu_buffer_rsrc_t rs, unsigned base, int lane, Waiter& W) {
    const unsigned probe = base - lane * 16u + (unsigned)(lane < N ? lane : N - 1) * 1024u + 1020u;
    for (;;) {
        unsigned v = 1u;
        if (lane == N - 1) v = __builtin_amdgcn_raw_buffer_load_b32(rs, probe, 0, 17);
        if (__all(v != 0u)) break;
        if (W.expired()) return;
        __builtin_amdgcn_s_sleep(POLL_SLEEP);
    }
}
template <int N, class OFF>
__device__ __forceinline__ void validate(u32x4* r, __amdgpu_buffer_rsrc_t rs, unsigned base, OFF off, Waiter& W) {
    unsigned m = 0xffffffffu;
#pragma unroll
    for (int f = 0; f < N; ++f) m = umin4(m, r[f]);
    while (__builtin_expect(!__all(m != 0u), 0)) {
        if (W.expired()) break;
#pragma unroll
        for (int f = 0; f < N; ++f) r[f] = __builtin_amdgcn_raw_buffer_load_b128(rs, base + off(f), 0, 17);
        m = 0xffffffffu;
#pragma unroll
        for (int f = 0; f < N; ++f) m = umin4(m, r[f]);
    }
}
struct Lin { __device__ __forceinline__ unsigned operator()(int f) const { return (unsigned)f * 1024u; } };

template <int N>
__device__ __forceinline__ void acquire(u32x4* r, __amdgpu_buffer_rsrc_t rs, unsigned base, int lane, Waiter& W) {
    wait_landed<N>(rs, base, lane, W);
#pragma unroll
    for (int f = 0; f < N; ++f) r[f] = __builtin_amdgcn_raw_buffer_load_b128(rs, base + f * 1024u, 0, 0);
    validate<N>(r, rs, base, Lin{}, W);
}

#define OUT_AUX 16
__device__ __forceinline__ f32x4 mfma(u32x4 a, u32x4 b, f32x4 c) {
    return __builtin_amdgcn_mfma_f32_16x16x32_f16(__builtin_bit_cast(half8, a), __builtin_bit_cast(half8, b), c, 0, 0, 0);
}

__device__ __forceinline__ unsigned wait_words(const char* p, unsigned call, Waiter& W) {
    u64 v;
    for (;;) {
        v = __hip_atomic_load((const u64*)p, __ATOMIC_RELAXED, __HIP_MEMORY_SCOPE_SYSTEM);
        if (__all((unsigned)(v >> 32) == call)) break;
        if (W.expired()) break;
        __builtin_amdgcn_s_sleep(2);
    }
    return (unsigned)v;
}

__global__ void __launch_bounds__(NTHREADS) gemm_ag_kernel(Params P) {
    __shared__ __attribute__((aligned(16))) float red[4 * 8 * 16 * 68];
    __shared__ unsigned s_m[4];
    __shared__ unsigned s_cnt[4];
    __shared__ int s_bad;

    const int g = blockIdx.x;
    const int wave = __builtin_amdgcn_readfirstlane(threadIdx.x >> 6);
    const int lane = threadIdx.x & 63;
    const int lg = lane >> 4, lc = lane & 15;
    const int rank = P.rank;
    char* const myws = P.myws;
    unsigned* ctr = (unsigned*)(myws + OFF_CTR + g * 128);
    TS_DECL;
    TS(0);
    const float* const px = P.x;
    const float* const pw = P.w;
    char* const wsx = P.ws[g < NPROD ? ((g >> 1) & 3) * 2 + (g & 1) : (g & 7)];
    asm volatile("" :: "s"(px), "s"(pw), "s"(myws), "s"(wsx), "s"(rank));
    const unsigned c0 = ctr_issue(ctr);
    const u64 tstart = __builtin_amdgcn_s_memrealtime();

    if (g < NPROD) {
        if (wave >= 4) {
            if (wave == 4) {
                char* const wsh = P.ws[g & 7];
                const unsigned call = ctr_wait(c0);
                if (lane < 4)
                    __hip_atomic_store((unsigned*)(wsh + OFF_HELLO + ((g >> 3) * 4 + lane) * 256 + rank * 4), call, __ATOMIC_RELAXED, __HIP_MEMORY_SCOPE_SYSTEM);
            }
            return;
        }
        const int half = g & 1, blk = g >> 1;
        const int t = (blk & 3) * 2 + half;
        const int rb16 = (blk >> 2) * 4 + wave;
        const int rowb = MS * t + 16 * rb16 + lc;
        f32x4 xb[8][2], wv[2][8];
        {
            const float* wsrc = pw + (size_t)(wave * 64 + lg * 8) * ND + half * 1024 + blk * 64 + lc * 4;
#pragma unroll
            for (int ks = 0; ks < 2; ++ks)
#pragma unroll
                for (int jj = 0; jj < 8; ++jj) wv[ks][jj] = *(const f32x4*)(wsrc + (size_t)(ks * 32 + jj) * ND);
        }
        asm volatile("" ::: "memory");
        const float* const xsrc = px + (size_t)rowb * KS + lg * 8;
        const unsigned call = ctr_wait(c0);
        const unsigned boff = OFF_DATA + (call % 3u) * BUF_BYTES;
        char* const dx = wsx + boff;
        const __amdgpu_buffer_rsrc_t rx = __builtin_amdgcn_make_buffer_rsrc(dx, 0, (int)BUF_BYTES, 0x00020000);
        {
            unsigned m = 0;
#pragma unroll
            for (int ks = 0; ks < 2; ++ks)
#pragma unroll
                for (int jj = 0; jj < 8; ++jj) m = umax4(m, wv[ks][jj]);
            m = wave_umax(m);
            if (lane == 0) s_m[wave] = m;
        }
        TS(1);
        asm volatile("s_waitcnt lgkmcnt(0)\n\ts_barrier" ::: "memory");
        unsigned ew;
        {
            unsigned m = s_m[0];
#pragma unroll
            for (int k = 1; k < 4; ++k) m = s_m[k] > m ? s_m[k] : m;
            ew = __builtin_amdgcn_readfirstlane(clamp_exp(m));
        }
        if (lane == 0) word_store(P.ws[wave * 2 + half] + boff + FW_OFF + (rank * 16 + blk) * 8, call, ew);
        __amdgpu_buffer_rsrc_t rw[4];
#pragma unroll
        for (int dd = 0; dd < 4; ++dd) {
            const int d = (((dd + wave + blk) & 3) << 1) + half;
            rw[dd] = __builtin_amdgcn_make_buffer_rsrc(P.ws[d] + boff, 0, (int)BUF_BYTES, 0x00020000);
        }
        unsigned eb = 0, pl = 0;
        int xi = 0;
        const unsigned t32 = (unsigned)tstart;
        TS(2);
        {
            const float mul = __builtin_bit_cast(float, (254u - ew) << 23);
            const unsigned off = WF_OFF + (unsigned)((rank * 16 + blk) * 32 + wave * 2) * 1024u + lane * 16u;
            const __amdgpu_buffer_rsrc_t rr = __builtin_amdgcn_make_buffer_rsrc(P.ws[((blk & 3) << 1) + half] + boff, 0, (int)BUF_BYTES, 0x00020000);
            const __amdgpu_buffer_rsrc_t rme = __builtin_amdgcn_make_buffer_rsrc(myws + boff, 0, (int)BUF_BYTES, 0x00020000);
            const unsigned roff = WF_OFF + (unsigned)(((2 * (blk & 3) + 1 - half) * 16 + (blk & 12) + (rank >> 1)) * 32 + 24 + wave * 2) * 1024u + lane * 16u;
            u32x4 rl[2];
            const auto pace = [&](bool same, int i, int n) {
                wait_vm_c(n + LW);
                const unsigned due = (unsigned)((same ? TS0 : TO0) + i * (same ? DS : DO) / 4);
                const unsigned now = (unsigned)__builtin_amdgcn_s_memrealtime() - t32;
                if (now <= due + LM) {
                    wait_vm_c(n);
                    if (i >= PFREE)
                        while ((unsigned)__builtin_amdgcn_s_memrealtime() - t32 < due) __builtin_amdgcn_s_sleep(1);
                }
            };
            const auto push = [&](auto relayed) {
                constexpr bool RL = decltype(relayed)::value == 1;
                constexpr bool SAME = decltype(relayed)::value == 0;
                constexpr int TH = SAME ? THRS : THR;
                int cnt = 0, extra = 0;
#pragma unroll
                for (int q = 0; q < 4; ++q) {
                    const int s = RL ? (q == RQ ? 3 : q < RQ ? q : q - 1) : q;
                    if (SKIP && q == 2) {
                        pl = __hip_atomic_load((const unsigned*)(myws + OFF_PLEDGE) + (2 * (blk >> 2) + 1 - half) * 16 + (blk & 3) * 4 + wave, __ATOMIC_RELAXED, __HIP_MEMORY_SCOPE_SYSTEM);
                        asm volatile("" ::: "memory");
                    }
                    u32x4 v[2];
#pragma unroll
                    for (int ks = 0; ks < 2; ++ks)
                        v[ks] = pack8(wv[ks][0][s], wv[ks][1][s], wv[ks][2][s], wv[ks][3][s], wv[ks][4][s], wv[ks][5][s], wv[ks][6][s], wv[ks][7][s], mul);
#pragma unroll
                    for (int dd = 0; dd < 4; ++dd)
#pragma unroll
                        for (int ks = 0; ks < 2; ++ks) {
                            if (RL && q == RQ && dd > 0) continue;
                            if (cnt >= TH) {
                                const int lo = cnt - TH > XL0 ? cnt - TH : XL0, hi = cnt < XL0 + 16 ? cnt : XL0 + 16;
                                pace(SAME, cnt + extra, TH - 1 + (hi > lo ? hi - lo : 0));
                            }
                            __builtin_amdgcn_raw_buffer_store_b128(v[ks], (RL && q == RQ) ? rr : rw[dd], off + s * 8192u + ks * 1024u, 0, ST_AUX);
                            asm volatile("" ::: "memory");
                            if (cnt >= XL0 && cnt < XL0 + 16) {
                                const int z = cnt - XL0;
                                xb[z >> 1][z & 1] = *(const f32x4*)(xsrc + (z >> 1) * 32 + (z & 1) * 4);
                                asm volatile("" ::: "memory");
                            }
                            ++cnt;
                        }
                    if (RELAY && SAME && q == RPOS) {
#pragma unroll
                        for (int f = 0; f < 2; ++f) rl[f] = __builtin_amdgcn_raw_buffer_load_b128(rme, roff + f * 1024u, 0, 17);
                        asm volatile("" ::: "memory");
                    }
                    if (RELAY && SAME && q == RPOS + 1 && (blk >> 2) < RB) {
                        TS(10);
                        asm volatile("" : "+v"(rl[0]), "+v"(rl[1]));
                        if (__builtin_expect(!__all(umin4(umin4(0xffffffffu, rl[0]), rl[1]) != 0u), 0)) {
                            Waiter W;
                            W.tmo = (unsigned*)(myws + OFF_TMO);
                            W.call = call;
                            W.t0 = __builtin_amdgcn_s_memrealtime();
                            W.ok = true;
                            do {
                                __builtin_amdgcn_s_sleep(POLL_SLEEP);
#pragma unroll
                                for (int f = 0; f < 2; ++f) rl[f] = __builtin_amdgcn_raw_buffer_load_b128(rme, roff + f * 1024u, 0, 17);
                            } while (!__all(umin4(umin4(0xffffffffu, rl[0]), rl[1]) != 0u) && !W.expired());
                        }
                        TS(9);
#pragma unroll
                        for (int dd = 0; dd < 4; ++dd) {
                            if ((((dd + wave + blk) & 3) << 1) + half == rank) continue;
#pragma unroll
                            for (int f = 0; f < 2; ++f) {
                                pace(true, cnt + (dd * 3 >> 1) + f, THRR - 1);
                                __builtin_amdgcn_raw_buffer_store_b128(rl[f], rw[dd], roff + f * 1024u, 0, ST_AUX);
                                asm volatile("" ::: "memory");
                            }
                        }
                    }
                    if (RELAY && SAME && q == RPOS + 1) extra = RB * 6 / 4;
                    xi = cnt + extra;
                    if (q == (RL ? 3 : 2)) {
                        unsigned m = 0;
#pragma unroll
                        for (int z = 0; z < 8; ++z) { m = umax4(m, xb[z][0]); m = umax4(m, xb[z][1]); }
                        eb = __builtin_amdgcn_readfirstlane(clamp_exp(wave_umax(m)));
                        if (lane == 0) word_store(dx + FX_OFF + (rank * 32 + 16 + rb16) * 8, call, eb);
                    }
                    TS(3 + q);
                }
            };
            if (RELAY && half != (rank & 1) && (blk >> 2) < RB) push(std::true_type{});
            else if (half != (rank & 1)) push(std::integral_constant<int, 2>{});
            else push(std::false_type{});
        }
        unsigned xdue = half != (rank & 1) ? (unsigned)(TO0 + xi * DO / 4 + TXO) : (unsigned)(TS0 + xi * DS / 4 + TXS);
        const unsigned xstep = half != (rank & 1) ? DXO : DXS;
        if (!SKIP || __builtin_amdgcn_readfirstlane(pl) != call) {
            const float mul = __builtin_bit_cast(float, (254u - eb) << 23);
            const unsigned off = XF_OFF + (unsigned)((rank * 32 + 16 + rb16) * 8) * 1024u + lane * 16u;
#pragma unroll
            for (int s = 0; s < 8; ++s) {
                const u32x4 v = pack8(xb[s][0][0], xb[s][0][1], xb[s][0][2], xb[s][0][3], xb[s][1][0], xb[s][1][1], xb[s][1][2], xb[s][1][3], mul);
                wait_vm<THRB - 1>();
                while ((unsigned)__builtin_amdgcn_s_memrealtime() - t32 < xdue) __builtin_amdgcn_s_sleep(1);
                xdue += xstep;
                __builtin_amdgcn_raw_buffer_store_b128(v, rx, off + s * 1024u, 0, ST_AUX);
                asm volatile("" ::: "memory");
            }
        }
        TS(7);
        wait_vm<0>();
        TS(8);
        if (threadIdx.x == 0) __hip_atomic_store(ctr, call, __ATOMIC_RELAXED, __HIP_MEMORY_SCOPE_AGENT);
        TS_FLUSH((half != (rank & 1) ? 12 : 8) + wave);
        return;
    }

    if (g >= NPROD + NFOR + NOWN + NJAN) {
        const unsigned call = ctr_wait(c0);
        const int dme = rank >> 1, par = rank & 1;
        const int hw = (g - (NPROD + NFOR + NOWN + NJAN)) * 8 + wave;
        const int hc = hw >> 4, hb = hw & 15;
        const unsigned boff = OFF_DATA + (call % 3u) * BUF_BYTES;
        Waiter W;
        W.tmo = (unsigned*)(myws + OFF_TMO);
        W.call = call;
        W.t0 = tstart;
        W.ok = true;
        unsigned late = 0;
        if (wave == 0) {
            while (__builtin_amdgcn_s_memrealtime() - tstart < (u64)TCHK) __builtin_amdgcn_s_sleep(1);
            for (;;) {
                const unsigned v = __hip_atomic_load((const unsigned*)(myws + OFF_HELLO + (g & 7) * 256) + (lane & 7), __ATOMIC_RELAXED, __HIP_MEMORY_SCOPE_SYSTEM);
                late = (unsigned)__builtin_amdgcn_ballot_w64((int)(v - call) < 0) & 0xffu;
                if (__builtin_popcount(late) <= NLATE) break;
                if (W.expired()) { late = 0; break; }
                __builtin_amdgcn_s_sleep(4);
            }
            if (lane == 0) s_m[0] = late;
        }
        __syncthreads();
        late = HELP ? __builtin_amdgcn_readfirstlane(s_m[0]) & ~(1u << rank) : 0u;
        late = (late >> (hc * 2)) & 3u;
        TS(1);
        if (SKIP && lane < 2 && ((late >> lane) & 1u))
            __hip_atomic_store((unsigned*)(P.ws[hc * 2 + lane] + OFF_PLEDGE + (rank * 16 + hb) * 4), call, __ATOMIC_RELAXED, __HIP_MEMORY_SCOPE_SYSTEM);
        const __amdgpu_buffer_rsrc_t rme = __builtin_amdgcn_make_buffer_rsrc(myws + boff, 0, (int)BUF_BYTES, 0x00020000);
        char* const dst = P.ws[2 * (hb >> 2) + 1 - par] + boff;
        const __amdgpu_buffer_rsrc_t rd = __builtin_amdgcn_make_buffer_rsrc(dst, 0, (int)BUF_BYTES, 0x00020000);
        u32x4 r[2][8];
        unsigned e[2] = {0u, 0u};
#pragma unroll
        for (int k = 0; k < 2; ++k) {
            if (!((late >> k) & 1u)) continue;
            const int hj = hc * 2 + k;
            const unsigned src = XF_OFF + (unsigned)((hj * 32 + hb) * 8) * 1024u + lane * 16u;
            wait_landed<8>(rme, src, lane, W);
#pragma unroll
            for (int f = 0; f < 8; ++f) r[k][f] = __builtin_amdgcn_raw_buffer_load_b128(rme, src + f * 1024u, 0, 17);
            e[k] = wait_words(myws + boff + FX_OFF + (hj * 32 + hb) * 8, call, W);
            validate<8>(r[k], rme, src, Lin{}, W);
        }
        TS(2);
        if (late) __syncthreads();
        TS(3);
#pragma unroll
        for (int k = 0; k < 2; ++k) {
            if (!((late >> k) & 1u) || !W.ok) continue;
            const unsigned slot = (unsigned)((hc * 2 + k) * 32 + 16 + dme * 4 + (hb & 3));
#pragma unroll
            for (int f = 0; f < 8; ++f) {
                if (THF < 16) wait_vm<THF - 1>();
                __builtin_amdgcn_raw_buffer_store_b128(r[k][f], rd, XF_OFF + slot * 8192u + lane * 16u + f * 1024u, 0, ST_AUX);
                asm volatile("" ::: "memory");
            }
            if (lane == 0) word_store(dst + FX_OFF + slot * 8u, call, e[k]);
        }
        TS(4);
        wait_vm<0>();
        TS(7);
        TS_FLUSH(wave);
        if (threadIdx.x == 0) __hip_atomic_store(ctr, call, __ATOMIC_RELAXED, __HIP_MEMORY_SCOPE_AGENT);
        return;
    }

    if (g >= NPROD + NFOR + NOWN) {
        const unsigned call = ctr_wait(c0);
        const __amdgpu_buffer_rsrc_t rz = __builtin_amdgcn_make_buffer_rsrc(myws + OFF_DATA + ((call + 2u) % 3u) * BUF_BYTES, 0, (int)ZERO_BYTES, 0x00020000);
        const unsigned zo = (unsigned)((g - NPROD - NFOR - NOWN) * 8 + wave) * 57344u + lane * 16u;
        {
            Waiter W;
            W.tmo = (unsigned*)(myws + OFF_TMO);
            W.call = call;
            W.t0 = tstart;
            W.ok = true;
            while (wave == 0) {
                const unsigned v = __hip_atomic_load((const unsigned*)(myws + OFF_HELLO + (8 + (g & 7)) * 256) + (lane & 7), __ATOMIC_RELAXED, __HIP_MEMORY_SCOPE_SYSTEM);
                if (__all((int)(v - call) >= 0) || W.expired()) break;
                __builtin_amdgcn_s_sleep(8);
            }
            __syncthreads();
        }
#pragma unroll 1
        for (int k = 0; k < 56 * ZREP; ++k) {
            wait_vm<ZTHR - 1>();
            __builtin_amdgcn_raw_buffer_store_b128((u32x4){0u, 0u, 0u, 0u}, rz, zo + (k % 56) * 1024u, 0, ZERO_AUX);
        }
        if (threadIdx.x == 0) __hip_atomic_store(ctr, call, __ATOMIC_RELAXED, __HIP_MEMORY_SCOPE_AGENT);
        return;
    }

    if (g >= NPROD + NFOR && wave == 0) {
        const int xt = g & 7, xrb = (g - NPROD - NFOR) >> 3;
        const int rowa = MS * (2 * (xrb >> 2) + 1 - (xt & 1)) + 64 * (xt >> 1) + 16 * (xrb & 3) + lc;
        f32x4 xa[8][2];
#pragma unroll
        for (int s = 0; s < 8; ++s) {
            const float* src = px + (size_t)rowa * KS + s * 32 + lg * 8;
            xa[s][0] = *(const f32x4*)src;
            xa[s][1] = *(const f32x4*)(src + 4);
        }
        const unsigned call = ctr_wait(c0);
        char* const dx = wsx + OFF_DATA + (call % 3u) * BUF_BYTES;
        const __amdgpu_buffer_rsrc_t rx = __builtin_amdgcn_make_buffer_rsrc(dx, 0, (int)BUF_BYTES, 0x00020000);
        unsigned m = 0;
#pragma unroll
        for (int s = 0; s < 8; ++s) { m = umax4(m, xa[s][0]); m = umax4(m, xa[s][1]); }
        const unsigned ea = __builtin_amdgcn_readfirstlane(clamp_exp(wave_umax(m)));
        const float mul = __builtin_bit_cast(float, (254u - ea) << 23);
        const unsigned off = XF_OFF + (unsigned)((rank * 32 + xrb) * 8) * 1024u + lane * 16u;
#pragma unroll
        for (int s = 0; s < 8; ++s) {
            const u32x4 v = pack8(xa[s][0][0], xa[s][0][1], xa[s][0][2], xa[s][0][3], xa[s][1][0], xa[s][1][1], xa[s][1][2], xa[s][1][3], mul);
            __builtin_amdgcn_raw_buffer_store_b128(v, rx, off + s * 1024u, 0, ST_AUX);
        }
        if (lane == 0) word_store(dx + FX_OFF + (rank * 32 + xrb) * 8, call, ea);
        TS(9);
    }
    const unsigned call = ctr_wait(c0);
    const unsigned boff = OFF_DATA + (call % 3u) * BUF_BYTES;
    const int j = wave;
    const int h = rank & 1;
    const __amdgpu_buffer_rsrc_t rs = __builtin_amdgcn_make_buffer_rsrc(myws + boff, 0, (int)BUF_BYTES, 0x00020000);
    const __amdgpu_buffer_rsrc_t ro = __builtin_amdgcn_make_buffer_rsrc(P.out, 0, MS * ND * 4, 0x00020000);
    if (threadIdx.x == 0) s_bad = 0;
    Waiter W;
    W.tmo = (unsigned*)(myws + OFF_TMO);
    W.call = call;
    W.t0 = __builtin_amdgcn_s_memrealtime();
    W.ok = true;
    const float qn = __builtin_nanf("");

    if (g < NPROD + NFOR) {
        const int cg = g - NPROD;
        const int blk = (cg & 7) * 2 + ((cg >> 3) & 1);
        const int rb64 = cg >> 4;
        if (threadIdx.x < 4) s_cnt[threadIdx.x] = 0;
        __syncthreads();
        u32x4 a[32];
        acquire<32>(a, rs, XF_OFF + (unsigned)((j * 32 + rb64 * 4) * 8) * 1024u + lane * 16u, lane, W);
        TS(1);
        const unsigned bbase = WF_OFF + (unsigned)((j * 16 + blk) * 32) * 1024u + lane * 16u;
        const int dst = 2 * rb64 + 1 - h;
        const __amdgpu_buffer_rsrc_t rd = __builtin_amdgcn_make_buffer_rsrc(P.ws[dst] + boff, 0, (int)BUF_BYTES, 0x00020000);
        const unsigned roff = RB_OFF + (unsigned)(((rank >> 1) * 16 + blk) * 4) * 4096u + threadIdx.x * 16u;
        int sc[4];
        unsigned pushed = 0;
        const auto reduce = [&](unsigned p) {
            const float* src = red + p * (8 * 16 * 68) + lc * 68 + (threadIdx.x >> 4) * 4;
            f32x4 sum = *(const f32x4*)src;
#pragma unroll
            for (int k = 1; k < 8; ++k) sum += *(const f32x4*)(src + k * (16 * 68));
            if (s_bad) sum = (f32x4){qn, qn, qn, qn};
            u32x4 v = __builtin_bit_cast(u32x4, sum);
#pragma unroll
            for (int k = 0; k < 4; ++k) v[k] = v[k] ? v[k] : 0x80000000u;
            return v;
        };
        const auto complete = [&](unsigned p) {
            const unsigned c = __builtin_amdgcn_readfirstlane(__hip_atomic_load(&s_cnt[p], __ATOMIC_RELAXED, __HIP_MEMORY_SCOPE_WORKGROUP));
            asm volatile("" ::: "memory");
            return c == 8u;
        };
#pragma unroll
        for (int s = 0; s < 4; ++s) {
            u32x4 b[8];
            wait_landed<8>(rs, bbase + s * 8192u, lane, W);
#pragma unroll
            for (int f = 0; f < 8; ++f) b[f] = __builtin_amdgcn_raw_buffer_load_b128(rs, bbase + s * 8192u + f * 1024u, 0, 0);
            if (s > 0) {
                asm volatile("" ::: "memory");
                u32x4 v = (u32x4){0u, 0u, 0u, 0u};
                unsigned off = 0x80000000u;
                if (wave < 4 && complete(pushed)) {
                    v = reduce(pushed);
                    off = roff + pushed * 4096u;
                    ++pushed;
                }
                __builtin_amdgcn_raw_buffer_store_b128(v, rd, off, 0, ST_AUX);
                asm volatile("" ::: "memory");
            }
            validate<8>(b, rs, bbase + s * 8192u, Lin{}, W);
            f32x4 acc[4];
#pragma unroll
            for (int rb = 0; rb < 4; ++rb) acc[rb] = (f32x4){0.f, 0.f, 0.f, 0.f};
#pragma unroll
            for (int k = 0; k < 8; ++k)
#pragma unroll
                for (int rb = 0; rb < 4; ++rb) acc[rb] = mfma(a[rb * 8 + k], b[k], acc[rb]);
            if (s == 0) {
                const unsigned wo = lane < 4 ? FX_OFF + (unsigned)(j * 32 + rb64 * 4 + lane) * 8u : FW_OFF + (unsigned)(j * 16 + blk) * 8u;
                const unsigned v = wait_words(myws + boff + wo, call, W);
#pragma unroll
                for (int rb = 0; rb < 4; ++rb) sc[rb] = (int)__builtin_amdgcn_readlane(v, rb) + (int)__builtin_amdgcn_readlane(v, 4) - 254;
            }
            float* const slab = red + (s * 8 + j) * (16 * 68);
#pragma unroll
            for (int rb = 0; rb < 4; ++rb) {
                f32x4 v;
#pragma unroll
                for (int r = 0; r < 4; ++r) v[r] = __builtin_ldexpf(acc[rb][r], sc[rb]);
                *(f32x4*)(slab + lc * 68 + rb * 16 + lg * 4) = v;
            }
            if (!W.ok && lane == 0) s_bad = 1;
            asm volatile("" ::: "memory");
            if (lane == 0) __hip_atomic_fetch_add(&s_cnt[s], 1u, __ATOMIC_RELAXED, __HIP_MEMORY_SCOPE_WORKGROUP);
            asm volatile("" ::: "memory");
            TS(2 + s);
        }
        if (wave < 4) {
            while (pushed < 4u) {
                while (!complete(pushed)) {
                    if (W.expired()) break;
                }
                u32x4 v = reduce(pushed);
                if (!W.ok) v = __builtin_bit_cast(u32x4, (f32x4){qn, qn, qn, qn});
                __builtin_amdgcn_raw_buffer_store_b128(v, rd, roff + pushed * 4096u, 0, ST_AUX);
                ++pushed;
            }
            TS(6);
        } else {
            const unsigned tid = threadIdx.x - 256u;
            const unsigned off = RB_OFF + (unsigned)((rb64 * 16 + blk) * 4) * 4096u + tid * 16u;
            u32x4 v[4];
            for (;;) {
#pragma unroll
                for (int s = 0; s < 4; ++s) v[s] = __builtin_amdgcn_raw_buffer_load_b128(rs, off + s * 4096u, 0, 17);
                if (__all(umin4(umin4(umin4(umin4(0xffffffffu, v[0]), v[1]), v[2]), v[3]) != 0u)) break;
                if (W.expired()) break;
                __builtin_amdgcn_s_sleep(HOT_SLEEP);
            }
            const unsigned o = (unsigned)((rb64 * 64 + (tid >> 4) * 4) * ND + (1 - h) * 1024 + blk * 64 + lc * 4) * 4u;
#pragma unroll
            for (int r = 0; r < 4; ++r) {
                f32x4 q = __builtin_bit_cast(f32x4, (u32x4){v[0][r], v[1][r], v[2][r], v[3][r]});
                if (!W.ok) q = (f32x4){qn, qn, qn, qn};
                __builtin_amdgcn_raw_buffer_store_b128(__builtin_bit_cast(u32x4, q), ro, o + (unsigned)r * (ND * 4u), 0, OUT_AUX);
            }
            TS(7);
        }
        if (threadIdx.x == 0) __hip_atomic_store(ctr, call, __ATOMIC_RELAXED, __HIP_MEMORY_SCOPE_AGENT);
        TS_FLUSH(wave);
        return;
    }

    {
        const int cg = g - NPROD - NFOR;
        const int blk = (cg & 7) * 2 + ((cg >> 3) & 1);
        const int rb32 = cg >> 4;
        u32x4 b[32];
        const unsigned bbase = WF_OFF + (unsigned)((j * 16 + blk) * 32) * 1024u + lane * 16u;
#pragma unroll
        for (int s = 0; s < 4; ++s) {
            acquire<8>(b + s * 8, rs, bbase + s * 8192u, lane, W);
            TS(1 + s);
        }
        int sc[2];
        {
            const unsigned wo = lane < 2 ? FX_OFF + (unsigned)(j * 32 + 16 + rb32 * 2 + lane) * 8u : FW_OFF + (unsigned)(j * 16 + blk) * 8u;
            const unsigned v = wait_words(myws + boff + wo, call, W);
#pragma unroll
            for (int rb = 0; rb < 2; ++rb) sc[rb] = (int)__builtin_amdgcn_readlane(v, rb) + (int)__builtin_amdgcn_readlane(v, 2) - 254;
        }
        const unsigned abase = XF_OFF + (unsigned)((j * 32 + 16 + rb32 * 2) * 8) * 1024u + lane * 16u;
        f32x4 acc[2][4];
#pragma unroll
        for (int rb = 0; rb < 2; ++rb)
#pragma unroll
            for (int s = 0; s < 4; ++s) acc[rb][s] = (f32x4){0.f, 0.f, 0.f, 0.f};
        for (;;) {
            unsigned v = 1u;
            if (lane == 63) v = __builtin_amdgcn_raw_buffer_load_b32(rs, abase + 8192u + 12u, 0, 17);
            if (__all(v != 0u)) break;
            if (W.expired()) break;
            __builtin_amdgcn_s_sleep(POLL_SLEEP);
        }
        TS(5);
        u32x4 a[2][8];
        const auto aoff = [](int f) { return (unsigned)((f >> 2) * 8192 + (f & 3) * 1024); };
#pragma unroll
        for (int c = 0; c < 2; ++c)
#pragma unroll
            for (int f = 0; f < 8; ++f) a[c][f] = __builtin_amdgcn_raw_buffer_load_b128(rs, abase + c * 4096u + aoff(f), 0, 17);
#pragma unroll
        for (int c = 0; c < 2; ++c) {
            validate<8>(a[c], rs, abase + c * 4096u, aoff, W);
#pragma unroll
            for (int ks = 0; ks < 4; ++ks)
#pragma unroll
                for (int rb = 0; rb < 2; ++rb)
#pragma unroll
                    for (int s = 0; s < 4; ++s) acc[rb][s] = mfma(a[c][rb * 4 + ks], b[s * 8 + c * 4 + ks], acc[rb][s]);
        }
        TS(6);
#pragma unroll
        for (int rb = 0; rb < 2; ++rb)
#pragma unroll
            for (int r = 0; r < 4; ++r) {
                f32x4 v;
#pragma unroll
                for (int s = 0; s < 4; ++s) v[s] = __builtin_ldexpf(acc[rb][s][r], sc[rb]);
                *(f32x4*)(red + (j * 32 + rb * 16 + lg * 4 + r) * 64 + lc * 4) = v;
            }
        if (!W.ok && lane == 0) s_bad = 1;
        __syncthreads();
        TS(7);
        const float* src = red + threadIdx.x * 4;
        f32x4 sum = *(const f32x4*)src;
#pragma unroll
        for (int k = 1; k < 8; ++k) sum += *(const f32x4*)(src + k * 2048);
        if (s_bad) sum = (f32x4){qn, qn, qn, qn};
        __builtin_amdgcn_raw_buffer_store_b128(__builtin_bit_cast(u32x4, sum), ro, (unsigned)((rb32 * 32 + (threadIdx.x >> 4)) * ND + h * 1024 + blk * 64 + lc * 4) * 4u, 0, OUT_AUX);
        if (threadIdx.x == 0) __hip_atomic_store(ctr, call, __ATOMIC_RELAXED, __HIP_MEMORY_SCOPE_AGENT);
        TS(8);
        TS_FLUSH(wave);
    }
}

}

extern "C" void kernel_launch(int rank, int nranks, ncclComm_t comm,
                              hipStream_t stream,
                              void* const* d_in, const int* in_sizes, int n_in,
                              void* d_out, int out_size,
                              const int* in_dtypes, int out_dtype,
                              void* const* d_out_all,
                              void* const* d_ws_all, size_t ws_bytes) {
    if (nranks != NR || (unsigned)rank >= (unsigned)NR || ws_bytes < (size_t)WS_NEED || n_in < 2 ||
        in_sizes[0] != 2048 * KS || in_sizes[1] != KS * ND || out_size != MS * ND) {
        fprintf(stderr, "kernel_launch: unexpected problem geometry\n");
        return;
    }
    Params P;
    P.x = (const float*)d_in[0];
    P.w = (const float*)d_in[1];
    P.out = (float*)d_out;
    for (int p = 0; p < NR; ++p) P.ws[p] = (char*)d_ws_all[p];
    P.rank = rank;
    P.myws = (char*)d_ws_all[rank];
    hipLaunchKernelGGL(gemm_ag_kernel, dim3(NWG), dim3(NTHREADS), 0, stream, P);
    HIP_CHECK(hipGetLastError());
}
```

```cpp
#include <hip/hip_runtime.h>
#include <rccl/rccl.h>
#include <cstdio>
#include <cstdint>
#include <utility>
#include <type_traits>

#define HIP_CHECK(expr) do { hipError_t _e = (expr); if (_e != hipSuccess) { \
    fprintf(stderr, "HIP error %s at %s:%d\n", hipGetErrorString(_e), __FILE__, __LINE__); \
    return; } } while (0)

#define ST_AUX 17
#define THR 4
#define THRB 8
#define RELAY 1
#define THRS 5
#define THRR THRS
#define ZREP 1
#define TXS 20
#define TXO 40
#define PMAP 0
#define RB 3
#define RQ 0
#define RPOS 1
#define XL0 0
#define TS0 280
#define DS 126
#define TO0 290
#define DO 124
#define DXS 24
#define DXO 45
#define LW 3
#define LM 30
#define PFREE 5
#define RTHR 2
#define THF 8
#define TCHK 350
#define LD0 100
#define PT0 1000
#define PT1 1280
#define PT2 1520
#define PS0 70
#define PS1 60
#define PS2 40
#define PREL 1
#define NLATE 3
#define HELP 2
#define SKIP 1
#define ZERO_AUX 17
#define ZTHR 4
#define POLL_SLEEP 1
#define HOT_SLEEP 1

#define TS_DECL
#define TS(slot) do { } while (0)
#define TS_FLUSH(w) do { } while (0)

namespace {

typedef _Float16 half8 __attribute__((ext_vector_type(8)));
typedef float f32x2 __attribute__((ext_vector_type(2)));
typedef float f32x4 __attribute__((ext_vector_type(4)));
typedef unsigned u32x4 __attribute__((ext_vector_type(4)));
typedef unsigned long long u64;

constexpr int NR = 8;
constexpr int ND = 2048;
constexpr int MS = 256;
constexpr int KS = 256;
constexpr int NPROD = 32;
constexpr int NFOR = 64;
constexpr int NOWN = 128;
constexpr int NJAN = 16;
constexpr int NHLP = 8;
constexpr int NWG = NPROD + NFOR + NOWN + NJAN + NHLP;
constexpr int NTHREADS = 512;

constexpr unsigned OFF_CTR = 0;
constexpr unsigned OFF_TMO = 65536;
constexpr unsigned OFF_HELLO = 65536 + 4096;
constexpr unsigned OFF_PLEDGE = 65536 + 8192;
constexpr unsigned OFF_DATA = 1u << 20;
constexpr unsigned XF_OFF = 0;
constexpr unsigned WF_OFF = 2u << 20;
constexpr unsigned RB_OFF = 6u << 20;
constexpr unsigned WL_OFF = 7u << 20;
constexpr unsigned ZERO_BYTES = 15u << 19;
constexpr unsigned FX_OFF = ZERO_BYTES;
constexpr unsigned FW_OFF = FX_OFF + 4096;
constexpr unsigned FWL_OFF = FW_OFF + 2048;
constexpr unsigned BUF_BYTES = ZERO_BYTES + 65536;
constexpr unsigned WS_NEED = OFF_DATA + 3 * BUF_BYTES;
constexpr u64 DEADLINE_TICKS = 50000000ull;

struct Params {
    const float* x;
    const float* w;
    float* out;
    char* myws;
    char* ws[NR];
    int rank;
};

template <int N> __device__ __forceinline__ void wait_vm() { asm volatile("s_waitcnt vmcnt(%0)" ::"n"(N) : "memory"); }

__device__ __forceinline__ void wait_vm_c(int n) {
    switch (n) {
        case 0: wait_vm<0>(); break;   case 1: wait_vm<1>(); break;   case 2: wait_vm<2>(); break;   case 3: wait_vm<3>(); break;
        case 4: wait_vm<4>(); break;   case 5: wait_vm<5>(); break;   case 6: wait_vm<6>(); break;   case 7: wait_vm<7>(); break;
        case 8: wait_vm<8>(); break;   case 9: wait_vm<9>(); break;   case 10: wait_vm<10>(); break; case 11: wait_vm<11>(); break;
        case 12: wait_vm<12>(); break; case 13: wait_vm<13>(); break; case 14: wait_vm<14>(); break; default: wait_vm<15>(); break;
    }
}

template <int CTRL> __device__ __forceinline__ unsigned dpp_umax(unsigned v) {
    const unsigned t = (unsigned)__builtin_amdgcn_update_dpp(0, (int)v, CTRL, 0xf, 0xf, false);
    return v > t ? v : t;
}
__device__ __forceinline__ unsigned wave_umax(unsigned v) {
    v = dpp_umax<0xB1>(v);
    v = dpp_umax<0x4E>(v);
    v = dpp_umax<0x141>(v);
    v = dpp_umax<0x140>(v);
    unsigned a = __builtin_amdgcn_readlane(v, 0), b = __builtin_amdgcn_readlane(v, 16);
    unsigned c = __builtin_amdgcn_readlane(v, 32), d = __builtin_amdgcn_readlane(v, 48);
    a = a > b ? a : b; c = c > d ? c : d;
    return a > c ? a : c;
}
__device__ __forceinline__ unsigned absbits(float v) { return __builtin_bit_cast(unsigned, v) & 0x7fffffffu; }
__device__ __forceinline__ unsigned umax4(unsigned m, f32x4 v) {
    unsigned a = absbits(v[0]), b = absbits(v[1]), c = absbits(v[2]), d = absbits(v[3]);
    a = a > b ? a : b; c = c > d ? c : d; a = a > c ? a : c;
    return m > a ? m : a;
}
__device__ __forceinline__ unsigned clamp_exp(unsigned m) {
    unsigned e = m >> 23;
    e = e < 1u ? 1u : e;
    return e > 253u ? 253u : e;
}
__device__ __forceinline__ void word_store(char* p, unsigned call, unsigned e) {
    __hip_atomic_store((u64*)p, ((u64)call << 32) | (u64)e, __ATOMIC_RELAXED, __HIP_MEMORY_SCOPE_SYSTEM);
}
__device__ __forceinline__ u32x4 pack8(float a0, float a1, float a2, float a3, float a4, float a5, float a6, float a7, float m) {
    half8 h;
    h[0] = (_Float16)(a0 * m); h[1] = (_Float16)(a1 * m); h[2] = (_Float16)(a2 * m); h[3] = (_Float16)(a3 * m);
    h[4] = (_Float16)(a4 * m); h[5] = (_Float16)(a5 * m); h[6] = (_Float16)(a6 * m); h[7] = (_Float16)(a7 * m);
    u32x4 v = __builtin_bit_cast(u32x4, h);
#pragma unroll
    for (int k = 0; k < 4; ++k) v[k] = v[k] ? v[k] : 0x8000u;
    return v;
}
__device__ __forceinline__ unsigned umin4(unsigned m, u32x4 v) {
    unsigned a = v[0] < v[1] ? v[0] : v[1], b = v[2] < v[3] ? v[2] : v[3];
    a = a < b ? a : b;
    return m < a ? m : a;
}
__device__ __forceinline__ unsigned ctr_issue(const unsigned* p) {
    unsigned v;
    asm volatile("s_load_dword %0, %1, 0x0" : "=s"(v) : "s"(p) : "memory");
    return v;
}
__device__ __forceinline__ unsigned ctr_wait(unsigned v) {
    unsigned r;
    asm volatile("s_waitcnt lgkmcnt(0)\n\ts_add_u32 %0, %1, 1" : "=s"(r) : "s"(v) : "memory", "scc");
    return r;
}

struct Waiter {
    unsigned* tmo;
    unsigned call;
    u64 t0;
    bool ok;
    __device__ __forceinline__ bool expired() {
        if (!ok) return true;
        if (__builtin_amdgcn_s_memrealtime() - t0 > DEADLINE_TICKS) {
            __hip_atomic_store(tmo, call, __ATOMIC_RELAXED, __HIP_MEMORY_SCOPE_AGENT);
            ok = false;
            return true;
        }
        return false;
    }
};

template <int N>
__device__ __forceinline__ void wait_landed(__amdgpu_buffer_rsrc_t rs, unsigned base, int lane, Waiter& W) {
    const unsigned probe = base - lane * 16u + (unsigned)(lane < N ? lane : N - 1) * 1024u + 1020u;
    for (;;) {
        unsigned v = 1u;
        if (lane == N - 1) v = __builtin_amdgcn_raw_buffer_load_b32(rs, probe, 0, 17);
        if (__all(v != 0u)) break;
        if (W.expired()) return;
        __builtin_amdgcn_s_sleep(POLL_SLEEP);
    }
}
template <int N, class OFF>
__device__ __forceinline__ void validate(u32x4* r, __amdgpu_buffer_rsrc_t rs, unsigned base, OFF off, Waiter& W) {
    unsigned m = 0xffffffffu;
#pragma unroll
    for (int f = 0; f < N; ++f) m = umin4(m, r[f]);
    while (__builtin_expect(!__all(m != 0u), 0)) {
        if (W.expired()) break;
#pragma unroll
        for (int f = 0; f < N; ++f) r[f] = __builtin_amdgcn_raw_buffer_load_b128(rs, base + off(f), 0, 17);
        m = 0xffffffffu;
#pragma unroll
        for (int f = 0; f < N; ++f) m = umin4(m, r[f]);
    }
}
struct Lin { __device__ __forceinline__ unsigned operator()(int f) const { return (unsigned)f * 1024u; } };

template <int N>
__device__ __forceinline__ void acquire(u32x4* r, __amdgpu_buffer_rsrc_t rs, unsigned base, int lane, Waiter& W) {
    wait_landed<N>(rs, base, lane, W);
#pragma unroll
    for (int f = 0; f < N; ++f) r[f] = __builtin_amdgcn_raw_buffer_load_b128(rs, base + f * 1024u, 0, 0);
    validate<N>(r, rs, base, Lin{}, W);
}

#define OUT_AUX 16
__device__ __forceinline__ f32x4 mfma(u32x4 a, u32x4 b, f32x4 c) {
    return __builtin_amdgcn_mfma_f32_16x16x32_f16(__builtin_bit_cast(half8, a), __builtin_bit_cast(half8, b), c, 0, 0, 0);
}

__device__ __forceinline__ unsigned wait_words(const char* p, unsigned call, Waiter& W) {
    u64 v;
    for (;;) {
        v = __hip_atomic_load((const u64*)p, __ATOMIC_RELAXED, __HIP_MEMORY_SCOPE_SYSTEM);
        if (__all((unsigned)(v >> 32) == call)) break;
        if (W.expired()) break;
        __builtin_amdgcn_s_sleep(2);
    }
    return (unsigned)v;
}

__global__ void __launch_bounds__(NTHREADS) gemm_ag_kernel(Params P) {
    __shared__ __attribute__((aligned(16))) float red[4 * 8 * 16 * 68];
    __shared__ unsigned s_m[12];
    __shared__ unsigned s_cnt[4];
    __shared__ int s_bad;

    const int g = blockIdx.x;
    const int wave = __builtin_amdgcn_readfirstlane(threadIdx.x >> 6);
    const int lane = threadIdx.x & 63;
    const int lg = lane >> 4, lc = lane & 15;
    const int rank = P.rank;
    char* const myws = P.myws;
    unsigned* ctr = (unsigned*)(myws + OFF_CTR + g * 128);
    TS_DECL;
    TS(0);
    const float* const px = P.x;
    const float* const pw = P.w;
    char* const wsx = P.ws[g < NPROD ? ((g >> 1) & 3) * 2 + (g & 1) : (g & 7)];
    asm volatile("" :: "s"(px), "s"(pw), "s"(myws), "s"(wsx), "s"(rank));
    const unsigned c0 = ctr_issue(ctr);
    const u64 tstart = __builtin_amdgcn_s_memrealtime();

    if (g < NPROD) {
        const int half = g & 1, blk = g >> 1;
        if (wave >= 4) {
            if (wave == 4) {
                char* const wsh = P.ws[g & 7];
                const unsigned call = ctr_wait(c0);
                if (lane < 4)
                    __hip_atomic_store((unsigned*)(wsh + OFF_HELLO + ((g >> 3) * 4 + lane) * 256 + rank * 4), call, __ATOMIC_RELAXED, __HIP_MEMORY_SCOPE_SYSTEM);
            }
            return;
        }
        const int t = (blk & 3) * 2 + half;
        const int rb16 = (blk >> 2) * 4 + wave;
        const int rowb = MS * t + 16 * rb16 + lc;
        f32x4 xb[8][2], wv[2][8];
        {
            const float* wsrc = pw + (size_t)(wave * 64 + lg * 8) * ND + half * 1024 + blk * 64 + lc * 4;
#pragma unroll
            for (int ks = 0; ks < 2; ++ks)
#pragma unroll
                for (int jj = 0; jj < 8; ++jj) wv[ks][jj] = *(const f32x4*)(wsrc + (size_t)(ks * 32 + jj) * ND);
        }
        asm volatile("" ::: "memory");
        const float* const xsrc = px + (size_t)rowb * KS + lg * 8;
        const unsigned call = ctr_wait(c0);
        const unsigned boff = OFF_DATA + (call % 3u) * BUF_BYTES;
        char* const dx = wsx + boff;
        const __amdgpu_buffer_rsrc_t rx = __builtin_amdgcn_make_buffer_rsrc(dx, 0, (int)BUF_BYTES, 0x00020000);
        {
            unsigned m = 0;
#pragma unroll
            for (int ks = 0; ks < 2; ++ks)
#pragma unroll
                for (int jj = 0; jj < 8; ++jj) m = umax4(m, wv[ks][jj]);
            m = wave_umax(m);
            if (lane == 0) s_m[wave] = m;
        }
        TS(1);
        asm volatile("s_waitcnt lgkmcnt(0)\n\ts_barrier" ::: "memory");
        unsigned ew;
        {
            unsigned m = s_m[0];
#pragma unroll
            for (int k = 1; k < 4; ++k) m = s_m[k] > m ? s_m[k] : m;
            ew = __builtin_amdgcn_readfirstlane(clamp_exp(m));
        }
        if (lane == 0) word_store(P.ws[wave * 2 + half] + boff + FW_OFF + (rank * 16 + blk) * 8, call, ew);
        __amdgpu_buffer_rsrc_t rw[4];
#pragma unroll
        for (int dd = 0; dd < 4; ++dd) {
            const int d = (((dd + wave + blk) & 3) << 1) + half;
            rw[dd] = __builtin_amdgcn_make_buffer_rsrc(P.ws[d] + boff, 0, (int)BUF_BYTES, 0x00020000);
        }
        unsigned eb = 0, pl = 0;
        int xi = 0;
        const unsigned t32 = (unsigned)tstart;
        TS(2);
        {
            const float mul = __builtin_bit_cast(float, (254u - ew) << 23);
            const unsigned off = WF_OFF + (unsigned)((rank * 16 + blk) * 32 + wave * 2) * 1024u + lane * 16u;
            const __amdgpu_buffer_rsrc_t rr = __builtin_amdgcn_make_buffer_rsrc(P.ws[((blk & 3) << 1) + half] + boff, 0, (int)BUF_BYTES, 0x00020000);
            const __amdgpu_buffer_rsrc_t rme = __builtin_amdgcn_make_buffer_rsrc(myws + boff, 0, (int)BUF_BYTES, 0x00020000);
            const unsigned roff = WF_OFF + (unsigned)(((2 * (blk & 3) + 1 - half) * 16 + (blk & 12) + (rank >> 1)) * 32 + 24 + wave * 2) * 1024u + lane * 16u;
            u32x4 rl[2];
            const auto pace = [&](bool same, int i, int n) {
                wait_vm_c(n + LW);
                const unsigned due = (unsigned)((same ? TS0 : TO0) + i * (same ? DS : DO) / 4);
                const unsigned now = (unsigned)__builtin_amdgcn_s_memrealtime() - t32;
                if (now <= due + LM) {
                    wait_vm_c(n);
                    if (i >= PFREE)
                        while ((unsigned)__builtin_amdgcn_s_memrealtime() - t32 < due) __builtin_amdgcn_s_sleep(1);
                }
            };
            const auto push = [&](auto relayed) {
                constexpr bool RL = decltype(relayed)::value == 1;
                constexpr bool SAME = decltype(relayed)::value == 0;
                constexpr int TH = SAME ? THRS : THR;
                int cnt = 0, extra = 0;
#pragma unroll
                for (int q = 0; q < 4; ++q) {
                    const int s = RL ? (q == RQ ? 3 : q < RQ ? q : q - 1) : q;
                    if (SKIP && q == 2) {
                        pl = __hip_atomic_load((const unsigned*)(myws + OFF_PLEDGE) + (2 * (blk >> 2) + 1 - half) * 16 + (blk & 3) * 4 + wave, __ATOMIC_RELAXED, __HIP_MEMORY_SCOPE_SYSTEM);
                        asm volatile("" ::: "memory");
                    }
                    u32x4 v[2];
#pragma unroll
                    for (int ks = 0; ks < 2; ++ks)
                        v[ks] = pack8(wv[ks][0][s], wv[ks][1][s], wv[ks][2][s], wv[ks][3][s], wv[ks][4][s], wv[ks][5][s], wv[ks][6][s], wv[ks][7][s], mul);
#pragma unroll
                    for (int dd = 0; dd < 4; ++dd)
#pragma unroll
                        for (int ks = 0; ks < 2; ++ks) {
                            if (RL && q == RQ && dd > 0) continue;
                            if (cnt >= TH) {
                                const int lo = cnt - TH > XL0 ? cnt - TH : XL0, hi = cnt < XL0 + 16 ? cnt : XL0 + 16;
                                pace(SAME, cnt + extra, TH - 1 + (hi > lo ? hi - lo : 0));
                            }
                            __builtin_amdgcn_raw_buffer_store_b128(v[ks], (RL && q == RQ) ? rr : rw[dd], off + s * 8192u + ks * 1024u, 0, ST_AUX);
                            asm volatile("" ::: "memory");
                            if (cnt >= XL0 && cnt < XL0 + 16) {
                                const int z = cnt - XL0;
                                xb[z >> 1][z & 1] = *(const f32x4*)(xsrc + (z >> 1) * 32 + (z & 1) * 4);
                                asm volatile("" ::: "memory");
                            }
                            ++cnt;
                        }
                    if (RELAY && SAME && q == RPOS) {
#pragma unroll
                        for (int f = 0; f < 2; ++f) rl[f] = __builtin_amdgcn_raw_buffer_load_b128(rme, roff + f * 1024u, 0, 17);
                        asm volatile("" ::: "memory");
                    }
                    if (RELAY && SAME && q == RPOS + 1 && (blk >> 2) < RB) {
                        TS(10);
                        asm volatile("" : "+v"(rl[0]), "+v"(rl[1]));
                        if (__builtin_expect(!__all(umin4(umin4(0xffffffffu, rl[0]), rl[1]) != 0u), 0)) {
                            Waiter W;
                            W.tmo = (unsigned*)(myws + OFF_TMO);
                            W.call = call;
                            W.t0 = __builtin_amdgcn_s_memrealtime();
                            W.ok = true;
                            do {
                                __builtin_amdgcn_s_sleep(POLL_SLEEP);
#pragma unroll
                                for (int f = 0; f < 2; ++f) rl[f] = __builtin_amdgcn_raw_buffer_load_b128(rme, roff + f * 1024u, 0, 17);
                            } while (!__all(umin4(umin4(0xffffffffu, rl[0]), rl[1]) != 0u) && !W.expired());
                        }
                        TS(9);
#pragma unroll
                        for (int dd = 0; dd < 4; ++dd) {
                            if ((((dd + wave + blk) & 3) << 1) + half == rank) continue;
#pragma unroll
                            for (int f = 0; f < 2; ++f) {
                                pace(true, cnt + (dd * 3 >> 1) + f, THRR - 1);
                                __builtin_amdgcn_raw_buffer_store_b128(rl[f], rw[dd], roff + f * 1024u, 0, ST_AUX);
                                asm volatile("" ::: "memory");
                            }
                        }
                    }
                    if (RELAY && SAME && q == RPOS + 1) extra = RB * 6 / 4;
                    xi = cnt + extra;
                    if (q == (RL ? 3 : 2)) {
                        unsigned m = 0;
#pragma unroll
                        for (int z = 0; z < 8; ++z) { m = umax4(m, xb[z][0]); m = umax4(m, xb[z][1]); }
                        eb = __builtin_amdgcn_readfirstlane(clamp_exp(wave_umax(m)));
                        if (lane == 0 && t != rank) word_store(dx + FX_OFF + (rank * 32 + 16 + rb16) * 8, call, eb);
                    }
                    TS(3 + q);
                }
            };
            if (RELAY && half != (rank & 1) && (blk >> 2) < RB) push(std::true_type{});
            else if (half != (rank & 1)) push(std::integral_constant<int, 2>{});
            else push(std::false_type{});
        }
        unsigned xdue = half != (rank & 1) ? (unsigned)(TO0 + xi * DO / 4 + TXO) : (unsigned)(TS0 + xi * DS / 4 + TXS);
        const unsigned xstep = half != (rank & 1) ? DXO : DXS;
        if (t != rank && (!SKIP || __builtin_amdgcn_readfirstlane(pl) != call)) {
            const float mul = __builtin_bit_cast(float, (254u - eb) << 23);
            const unsigned off = XF_OFF + (unsigned)((rank * 32 + 16 + rb16) * 8) * 1024u + lane * 16u;
#pragma unroll
            for (int s = 0; s < 8; ++s) {
                const u32x4 v = pack8(xb[s][0][0], xb[s][0][1], xb[s][0][2], xb[s][0][3], xb[s][1][0], xb[s][1][1], xb[s][1][2], xb[s][1][3], mul);
                wait_vm<THRB - 1>();
                while ((unsigned)__builtin_amdgcn_s_memrealtime() - t32 < xdue) __builtin_amdgcn_s_sleep(1);
                xdue += xstep;
                __builtin_amdgcn_raw_buffer_store_b128(v, rx, off + s * 1024u, 0, ST_AUX);
                asm volatile("" ::: "memory");
            }
        }
        TS(7);
        wait_vm<0>();
        TS(8);
        if (threadIdx.x == 0) __hip_atomic_store(ctr, call, __ATOMIC_RELAXED, __HIP_MEMORY_SCOPE_AGENT);
        TS_FLUSH((half != (rank & 1) ? 12 : 8) + wave);
        return;
    }

    if (g >= NPROD + NFOR + NOWN + NJAN) {
        const int dme = rank >> 1, par = rank & 1;
        const int hw = (g - (NPROD + NFOR + NOWN + NJAN)) * 8 + wave;
        const int hc = hw >> 4, hb = hw & 15;
        f32x4 wv[2][8];
        {
            if (LD0) while (__builtin_amdgcn_s_memrealtime() - tstart < (u64)LD0) __builtin_amdgcn_s_sleep(1);
            const float* wsrc = pw + (size_t)((wave & 3) * 64 + lg * 8) * ND + (1 - par) * 1024 + (hw >> 2) * 64 + lc * 4;
#pragma unroll
            for (int ks = 0; ks < 2; ++ks)
#pragma unroll
                for (int jj = 0; jj < 8; ++jj) wv[ks][jj] = *(const f32x4*)(wsrc + (size_t)(ks * 32 + jj) * ND);
        }
        const unsigned call = ctr_wait(c0);
        const unsigned boff = OFF_DATA + (call % 3u) * BUF_BYTES;
        const __amdgpu_buffer_rsrc_t rme = __builtin_amdgcn_make_buffer_rsrc(myws + boff, 0, (int)BUF_BYTES, 0x00020000);
        {
            unsigned m = 0;
#pragma unroll
            for (int ks = 0; ks < 2; ++ks)
#pragma unroll
                for (int jj = 0; jj < 8; ++jj) m = umax4(m, wv[ks][jj]);
            m = wave_umax(m);
            if (lane == 0) s_m[4 + wave] = m;
            TS(5);
            __syncthreads();
            m = s_m[4 + (wave & 4)];
#pragma unroll
            for (int k = 1; k < 4; ++k) m = s_m[4 + (wave & 4) + k] > m ? s_m[4 + (wave & 4) + k] : m;
            const unsigned ew = __builtin_amdgcn_readfirstlane(clamp_exp(m));
            const float mul = __builtin_bit_cast(float, (254u - ew) << 23);
            const unsigned off = WL_OFF + (unsigned)((hw >> 2) * 32 + (wave & 3) * 2) * 1024u + lane * 16u;
#pragma unroll
            for (int s = 0; s < 4; ++s)
#pragma unroll
                for (int ks = 0; ks < 2; ++ks) {
                    const u32x4 v = pack8(wv[ks][0][s], wv[ks][1][s], wv[ks][2][s], wv[ks][3][s], wv[ks][4][s], wv[ks][5][s], wv[ks][6][s], wv[ks][7][s], mul);
                    __builtin_amdgcn_raw_buffer_store_b128(v, rme, off + s * 8192u + ks * 1024u, 0, ST_AUX);
                }
            if ((threadIdx.x & 255) == 0) word_store(myws + boff + FWL_OFF + (hw >> 2) * 8, call, ew);
            TS(6);
        }
        Waiter W;
        W.tmo = (unsigned*)(myws + OFF_TMO);
        W.call = call;
        W.t0 = tstart;
        W.ok = true;
        unsigned late = 0;
        if (wave == 0) {
            while (__builtin_amdgcn_s_memrealtime() - tstart < (u64)TCHK) __builtin_amdgcn_s_sleep(1);
            for (;;) {
                const unsigned v = __hip_atomic_load((const unsigned*)(myws + OFF_HELLO + (g & 7) * 256) + (lane & 7), __ATOMIC_RELAXED, __HIP_MEMORY_SCOPE_SYSTEM);
                late = (unsigned)__builtin_amdgcn_ballot_w64((int)(v - call) < 0) & 0xffu;
                if (__builtin_popcount(late) <= NLATE) break;
                if (W.expired()) { late = 0; break; }
                __builtin_amdgcn_s_sleep(4);
            }
            if (lane == 0) s_m[0] = late;
        }
        __syncthreads();
        late = HELP ? __builtin_amdgcn_readfirstlane(s_m[0]) & ~(1u << rank) : 0u;
        late &= ~(1u << (2 * (hb >> 2) + 1 - par));
        late = (late >> (hc * 2)) & 3u;
        TS(1);
        if (SKIP && lane < 2 && ((late >> lane) & 1u))
            __hip_atomic_store((unsigned*)(P.ws[hc * 2 + lane] + OFF_PLEDGE + (rank * 16 + hb) * 4), call, __ATOMIC_RELAXED, __HIP_MEMORY_SCOPE_SYSTEM);
        char* const dst = P.ws[2 * (hb >> 2) + 1 - par] + boff;
        const __amdgpu_buffer_rsrc_t rd = __builtin_amdgcn_make_buffer_rsrc(dst, 0, (int)BUF_BYTES, 0x00020000);
        u32x4 r[2][8];
        unsigned e[2] = {0u, 0u};
#pragma unroll
        for (int k = 0; k < 2; ++k) {
            if (!((late >> k) & 1u)) continue;
            const int hj = hc * 2 + k;
            const unsigned src = XF_OFF + (unsigned)((hj * 32 + hb) * 8) * 1024u + lane * 16u;
            wait_landed<8>(rme, src, lane, W);
#pragma unroll
            for (int f = 0; f < 8; ++f) r[k][f] = __builtin_amdgcn_raw_buffer_load_b128(rme, src + f * 1024u, 0, 17);
            e[k] = wait_words(myws + boff + FX_OFF + (hj * 32 + hb) * 8, call, W);
            validate<8>(r[k], rme, src, Lin{}, W);
        }
        TS(2);
        if (late) __syncthreads();
        TS(3);
#pragma unroll
        for (int k = 0; k < 2; ++k) {
            if (!((late >> k) & 1u) || !W.ok) continue;
            const unsigned slot = (unsigned)((hc * 2 + k) * 32 + 16 + dme * 4 + (hb & 3));
#pragma unroll
            for (int f = 0; f < 8; ++f) {
                if (THF < 16) wait_vm<THF - 1>();
                __builtin_amdgcn_raw_buffer_store_b128(r[k][f], rd, XF_OFF + slot * 8192u + lane * 16u + f * 1024u, 0, ST_AUX);
                asm volatile("" ::: "memory");
            }
            if (lane == 0) word_store(dst + FX_OFF + slot * 8u, call, e[k]);
        }
        TS(4);
        wait_vm<0>();
        TS(7);
        TS_FLUSH(wave);
        if (threadIdx.x == 0) __hip_atomic_store(ctr, call, __ATOMIC_RELAXED, __HIP_MEMORY_SCOPE_AGENT);
        return;
    }

    if (g >= NPROD + NFOR + NOWN) {
        const unsigned call = ctr_wait(c0);
        const __amdgpu_buffer_rsrc_t rz = __builtin_amdgcn_make_buffer_rsrc(myws + OFF_DATA + ((call + 2u) % 3u) * BUF_BYTES, 0, (int)ZERO_BYTES, 0x00020000);
        const unsigned zo = (unsigned)((g - NPROD - NFOR - NOWN) * 8 + wave) * 61440u + lane * 16u;
        {
            Waiter W;
            W.tmo = (unsigned*)(myws + OFF_TMO);
            W.call = call;
            W.t0 = tstart;
            W.ok = true;
            while (wave == 0) {
                const unsigned v = __hip_atomic_load((const unsigned*)(myws + OFF_HELLO + (8 + (g & 7)) * 256) + (lane & 7), __ATOMIC_RELAXED, __HIP_MEMORY_SCOPE_SYSTEM);
                if (__all((int)(v - call) >= 0) || W.expired()) break;
                __builtin_amdgcn_s_sleep(8);
            }
            __syncthreads();
        }
#pragma unroll 1
        for (int k = 0; k < 60 * ZREP; ++k) {
            wait_vm<ZTHR - 1>();
            __builtin_amdgcn_raw_buffer_store_b128((u32x4){0u, 0u, 0u, 0u}, rz, zo + (k % 60) * 1024u, 0, ZERO_AUX);
        }
        if (threadIdx.x == 0) __hip_atomic_store(ctr, call, __ATOMIC_RELAXED, __HIP_MEMORY_SCOPE_AGENT);
        return;
    }

    if (g >= NPROD + NFOR && wave == 0) {
        const int xt = g & 7, xrb = (g - NPROD - NFOR) >> 3;
        const int xown = 2 * (xrb >> 2) + 1 - (xt & 1);
        const int rowa = MS * xown + 64 * (xt >> 1) + 16 * (xrb & 3) + lc;
        const int xslot = rank * 32 + (xown == rank ? 16 + 4 * (xt >> 1) + (xrb & 3) : xrb);
        f32x4 xa[8][2];
#pragma unroll
        for (int s = 0; s < 8; ++s) {
            const float* src = px + (size_t)rowa * KS + s * 32 + lg * 8;
            xa[s][0] = *(const f32x4*)src;
            xa[s][1] = *(const f32x4*)(src + 4);
        }
        const unsigned call = ctr_wait(c0);
        char* const dx = (xown == rank ? myws : wsx) + OFF_DATA + (call % 3u) * BUF_BYTES;
        const __amdgpu_buffer_rsrc_t rx = __builtin_amdgcn_make_buffer_rsrc(dx, 0, (int)BUF_BYTES, 0x00020000);
        unsigned m = 0;
#pragma unroll
        for (int s = 0; s < 8; ++s) { m = umax4(m, xa[s][0]); m = umax4(m, xa[s][1]); }
        const unsigned ea = __builtin_amdgcn_readfirstlane(clamp_exp(wave_umax(m)));
        const float mul = __builtin_bit_cast(float, (254u - ea) << 23);
        const unsigned off = XF_OFF + (unsigned)(xslot * 8) * 1024u + lane * 16u;
#pragma unroll
        for (int s = 0; s < 8; ++s) {
            const u32x4 v = pack8(xa[s][0][0], xa[s][0][1], xa[s][0][2], xa[s][0][3], xa[s][1][0], xa[s][1][1], xa[s][1][2], xa[s][1][3], mul);
            __builtin_amdgcn_raw_buffer_store_b128(v, rx, off + s * 1024u, 0, ST_AUX);
        }
        if (lane == 0) word_store(dx + FX_OFF + xslot * 8, call, ea);
        TS(9);
    }
    const unsigned call = ctr_wait(c0);
    const unsigned boff = OFF_DATA + (call % 3u) * BUF_BYTES;
    const int j = wave;
    const int h = rank & 1;
    const __amdgpu_buffer_rsrc_t rs = __builtin_amdgcn_make_buffer_rsrc(myws + boff, 0, (int)BUF_BYTES, 0x00020000);
    const __amdgpu_buffer_rsrc_t ro = __builtin_amdgcn_make_buffer_rsrc(P.out, 0, MS * ND * 4, 0x00020000);
    if (threadIdx.x == 0) s_bad = 0;
    Waiter W;
    W.tmo = (unsigned*)(myws + OFF_TMO);
    W.call = call;
    W.t0 = __builtin_amdgcn_s_memrealtime();
    W.ok = true;
    const float qn = __builtin_nanf("");

    if (g < NPROD + NFOR) {
        const int cg = g - NPROD;
        const int blk = (cg & 7) * 2 + ((cg >> 3) & 1);
        const int rb64 = cg >> 4;
        if (threadIdx.x < 4) s_cnt[threadIdx.x] = 0;
        __syncthreads();
        const int dst = 2 * rb64 + 1 - h;
        const bool loc = j == dst;
        const int aslot = loc ? rank * 32 + 16 + rb64 * 4 : j * 32 + rb64 * 4;
        const int slabi = loc ? 7 : j < dst ? j : j - 1;
        u32x4 a[32];
        acquire<32>(a, rs, XF_OFF + (unsigned)(aslot * 8) * 1024u + lane * 16u, lane, W);
        TS(1);
        const unsigned bbase = (loc ? WL_OFF + (unsigned)(blk * 32) * 1024u : WF_OFF + (unsigned)((j * 16 + blk) * 32) * 1024u) + lane * 16u;
        const __amdgpu_buffer_rsrc_t rd = __builtin_amdgcn_make_buffer_rsrc(P.ws[dst] + boff, 0, (int)BUF_BYTES, 0x00020000);
        const unsigned roff = RB_OFF + (unsigned)(((rank >> 1) * 16 + blk) * 4) * 4096u + lane * 16u;
        int sc[4];
        const auto reduce = [&](unsigned p, int qt) {
            const float* src = red + p * (8 * 16 * 68) + lc * 68 + (qt * 4 + lg) * 4;
            f32x4 sum = *(const f32x4*)src;
#pragma unroll
            for (int k = 1; k < 7; ++k) sum += *(const f32x4*)(src + k * (16 * 68));
            if (s_bad) sum = (f32x4){qn, qn, qn, qn};
            u32x4 v = __builtin_bit_cast(u32x4, sum);
#pragma unroll
            for (int k = 0; k < 4; ++k) v[k] = v[k] ? v[k] : 0x80000000u;
            return v;
        };
        const auto complete = [&](unsigned p) {
            const unsigned c = __builtin_amdgcn_readfirstlane(__hip_atomic_load(&s_cnt[p], __ATOMIC_RELAXED, __HIP_MEMORY_SCOPE_WORKGROUP));
            asm volatile("" ::: "memory");
            return (c & 255u) == 7u;
        };
#pragma unroll
        for (int s = 0; s < 4; ++s) {
            u32x4 b[8];
            if (s == 0 || !loc) wait_landed<8>(rs, bbase + (loc ? 3 : s) * 8192u, lane, W);
#pragma unroll
            for (int f = 0; f < 8; ++f) b[f] = __builtin_amdgcn_raw_buffer_load_b128(rs, bbase + s * 8192u + f * 1024u, 0, 0);
            validate<8>(b, rs, bbase + s * 8192u, Lin{}, W);
            f32x4 acc[4];
#pragma unroll
            for (int rb = 0; rb < 4; ++rb) acc[rb] = (f32x4){0.f, 0.f, 0.f, 0.f};
#pragma unroll
            for (int k = 0; k < 8; ++k)
#pragma unroll
                for (int rb = 0; rb < 4; ++rb) acc[rb] = mfma(a[rb * 8 + k], b[k], acc[rb]);
            if (s == 0) {
                const unsigned wo = lane < 4 ? FX_OFF + (unsigned)(aslot + lane) * 8u : loc ? FWL_OFF + (unsigned)blk * 8u : FW_OFF + (unsigned)(j * 16 + blk) * 8u;
                const unsigned v = wait_words(myws + boff + wo, call, W);
#pragma unroll
                for (int rb = 0; rb < 4; ++rb) sc[rb] = (int)__builtin_amdgcn_readlane(v, rb) + (int)__builtin_amdgcn_readlane(v, 4) - 254;
            }
            float* const slab = red + (s * 8 + slabi) * (16 * 68);
#pragma unroll
            for (int rb = 0; rb < 4; ++rb) {
                f32x4 v;
#pragma unroll
                for (int r = 0; r < 4; ++r) v[r] = __builtin_ldexpf(acc[rb][r], sc[rb]);
                *(f32x4*)(slab + lc * 68 + rb * 16 + lg * 4) = v;
            }
            if (!W.ok && lane == 0) s_bad = 1;
            asm volatile("" ::: "memory");
            if (lane == 0) __hip_atomic_fetch_add(&s_cnt[s], loc ? 256u : 1u, __ATOMIC_RELAXED, __HIP_MEMORY_SCOPE_WORKGROUP);
            asm volatile("" ::: "memory");
            TS(2 + s);
            if (loc && s == 0) TS(8);
            if (loc && s == 3) TS(9);
        }
        if (loc) {
#pragma unroll 1
            for (unsigned p = 0; p < 3u; ++p) {
                while (!complete(p)) {
                    if (W.expired()) break;
                    __builtin_amdgcn_s_sleep(1);
                }
#pragma unroll
                for (int qt = 0; qt < 4; ++qt) {
                    u32x4 v = reduce(p, qt);
                    if (!W.ok) v = __builtin_bit_cast(u32x4, (f32x4){qn, qn, qn, qn});
                    {
                        const unsigned ps = p == 0u ? PS0 : p == 1u ? PS1 : PS2;
                        const unsigned due = (p == 0u ? PT0 : p == 1u ? PT1 : PT2) + qt * ps + (cg & 15) * ps / 16u;
                        while ((unsigned)(__builtin_amdgcn_s_memrealtime() - tstart) < due && !(PREL && complete(3u))) __builtin_amdgcn_s_sleep(1);
                    }
                    __builtin_amdgcn_raw_buffer_store_b128(v, rd, roff + p * 4096u + qt * 1024u, 0, ST_AUX);
                }
            }
            TS(10);
        }
        if (slabi < 4) {
            while (!complete(3u)) {
                if (W.expired()) break;
            }
            u32x4 v = reduce(3u, slabi);
            if (!W.ok) v = __builtin_bit_cast(u32x4, (f32x4){qn, qn, qn, qn});
            __builtin_amdgcn_raw_buffer_store_b128(v, rd, roff + 3u * 4096u + slabi * 1024u, 0, ST_AUX);
            TS(6);
        } else {
            const unsigned tid = (unsigned)(slabi - 4) * 64u + lane;
            const unsigned off = RB_OFF + (unsigned)((rb64 * 16 + blk) * 4) * 4096u + tid * 16u;
            u32x4 v[4];
            f32x4 lp[4];
            while (__builtin_amdgcn_readfirstlane(__hip_atomic_load(&s_cnt[3], __ATOMIC_RELAXED, __HIP_MEMORY_SCOPE_WORKGROUP)) < 256u) {
                if (W.expired()) break;
            }
            asm volatile("" ::: "memory");
#pragma unroll
            for (int s = 0; s < 4; ++s) lp[s] = *(const f32x4*)(red + (s * 8 + 7) * (16 * 68) + lc * 68 + (tid >> 4) * 4);
            for (;;) {
#pragma unroll
                for (int s = 0; s < 4; ++s) v[s] = __builtin_amdgcn_raw_buffer_load_b128(rs, off + s * 4096u, 0, 17);
                if (__all(umin4(umin4(umin4(umin4(0xffffffffu, v[0]), v[1]), v[2]), v[3]) != 0u)) break;
                if (W.expired()) break;
                __builtin_amdgcn_s_sleep(HOT_SLEEP);
            }
            const unsigned o = (unsigned)((rb64 * 64 + (tid >> 4) * 4) * ND + (1 - h) * 1024 + blk * 64 + lc * 4) * 4u;
#pragma unroll
            for (int r = 0; r < 4; ++r) {
                f32x4 q = __builtin_bit_cast(f32x4, (u32x4){v[0][r], v[1][r], v[2][r], v[3][r]});
                q += (f32x4){lp[0][r], lp[1][r], lp[2][r], lp[3][r]};
                if (!W.ok || s_bad) q = (f32x4){qn, qn, qn, qn};
                __builtin_amdgcn_raw_buffer_store_b128(__builtin_bit_cast(u32x4, q), ro, o + (unsigned)r * (ND * 4u), 0, OUT_AUX);
            }
            TS(7);
        }
        if (threadIdx.x == 0) __hip_atomic_store(ctr, call, __ATOMIC_RELAXED, __HIP_MEMORY_SCOPE_AGENT);
        TS_FLUSH(wave);
        return;
    }

    {
        const int cg = g - NPROD - NFOR;
        const int blk = (cg & 7) * 2 + ((cg >> 3) & 1);
        const int rb32 = cg >> 4;
        u32x4 b[32];
        const unsigned bbase = WF_OFF + (unsigned)((j * 16 + blk) * 32) * 1024u + lane * 16u;
#pragma unroll
        for (int s = 0; s < 4; ++s) {
            acquire<8>(b + s * 8, rs, bbase + s * 8192u, lane, W);
            TS(1 + s);
        }
        int sc[2];
        {
            const unsigned wo = lane < 2 ? FX_OFF + (unsigned)(j * 32 + 16 + rb32 * 2 + lane) * 8u : FW_OFF + (unsigned)(j * 16 + blk) * 8u;
            const unsigned v = wait_words(myws + boff + wo, call, W);
#pragma unroll
            for (int rb = 0; rb < 2; ++rb) sc[rb] = (int)__builtin_amdgcn_readlane(v, rb) + (int)__builtin_amdgcn_readlane(v, 2) - 254;
        }
        const unsigned abase = XF_OFF + (unsigned)((j * 32 + 16 + rb32 * 2) * 8) * 1024u + lane * 16u;
        f32x4 acc[2][4];
#pragma unroll
        for (int rb = 0; rb < 2; ++rb)
#pragma unroll
            for (int s = 0; s < 4; ++s) acc[rb][s] = (f32x4){0.f, 0.f, 0.f, 0.f};
        for (;;) {
            unsigned v = 1u;
            if (lane == 63) v = __builtin_amdgcn_raw_buffer_load_b32(rs, abase + 8192u + 12u, 0, 17);
            if (__all(v != 0u)) break;
            if (W.expired()) break;
            __builtin_amdgcn_s_sleep(POLL_SLEEP);
        }
        TS(5);
        u32x4 a[2][8];
        const auto aoff = [](int f) { return (unsigned)((f >> 2) * 8192 + (f & 3) * 1024); };
#pragma unroll
        for (int c = 0; c < 2; ++c)
#pragma unroll
            for (int f = 0; f < 8; ++f) a[c][f] = __builtin_amdgcn_raw_buffer_load_b128(rs, abase + c * 4096u + aoff(f), 0, 17);
#pragma unroll
        for (int c = 0; c < 2; ++c) {
            validate<8>(a[c], rs, abase + c * 4096u, aoff, W);
#pragma unroll
            for (int ks = 0; ks < 4; ++ks)
#pragma unroll
                for (int rb = 0; rb < 2; ++rb)
#pragma unroll
                    for (int s = 0; s < 4; ++s) acc[rb][s] = mfma(a[c][rb * 4 + ks], b[s * 8 + c * 4 + ks], acc[rb][s]);
        }
        TS(6);
#pragma unroll
        for (int rb = 0; rb < 2; ++rb)
#pragma unroll
            for (int r = 0; r < 4; ++r) {
                f32x4 v;
#pragma unroll
                for (int s = 0; s < 4; ++s) v[s] = __builtin_ldexpf(acc[rb][s][r], sc[rb]);
                *(f32x4*)(red + (j * 32 + rb * 16 + lg * 4 + r) * 64 + lc * 4) = v;
            }
        if (!W.ok && lane == 0) s_bad = 1;
        __syncthreads();
        TS(7);
        const float* src = red + threadIdx.x * 4;
        f32x4 sum = *(const f32x4*)src;
#pragma unroll
        for (int k = 1; k < 8; ++k) sum += *(const f32x4*)(src + k * 2048);
        if (s_bad) sum = (f32x4){qn, qn, qn, qn};
        __builtin_amdgcn_raw_buffer_store_b128(__builtin_bit_cast(u32x4, sum), ro, (unsigned)((rb32 * 32 + (threadIdx.x >> 4)) * ND + h * 1024 + blk * 64 + lc * 4) * 4u, 0, OUT_AUX);
        if (threadIdx.x == 0) __hip_atomic_store(ctr, call, __ATOMIC_RELAXED, __HIP_MEMORY_SCOPE_AGENT);
        TS(8);
        TS_FLUSH(wave);
    }
}

}

extern "C" void kernel_launch(int rank, int nranks, ncclComm_t comm,
                              hipStream_t stream,
                              void* const* d_in, const int* in_sizes, int n_in,
                              void* d_out, int out_size,
                              const int* in_dtypes, int out_dtype,
                              void* const* d_out_all,
                              void* const* d_ws_all, size_t ws_bytes) {
    if (nranks != NR || (unsigned)rank >= (unsigned)NR || ws_bytes < (size_t)WS_NEED || n_in < 2 ||
        in_sizes[0] != 2048 * KS || in_sizes[1] != KS * ND || out_size != MS * ND) {
        fprintf(stderr, "kernel_launch: unexpected problem geometry\n");
        return;
    }
    Params P;
    P.x = (const float*)d_in[0];
    P.w = (const float*)d_in[1];
    P.out = (float*)d_out;
    for (int p = 0; p < NR; ++p) P.ws[p] = (char*)d_ws_all[p];
    P.rank = rank;
    P.myws = (char*)d_ws_all[rank];
    hipLaunchKernelGGL(gemm_ag_kernel, dim3(NWG), dim3(NTHREADS), 0, stream, P);
    HIP_CHECK(hipGetLastError());
}
```
